```python
import numpy as np

kernel_name = "gpu_gemm_rs_m4096_k4096_n4096_w8a8_dqsilu"


ABSMAX_RTOL = 0.001

COMPUTE_DTYPE = "int8"
EPILOGUE = "dequant_silu"
IN_QUANT = "int8"
OUT_QUANT = None
IN_DTYPES = {"x": "int8", "w_mat": "int8",
             "scale_x": "float32", "scale_w": "float32"}
WORLD_SIZE = 8

def setup_inputs():

    x = np.random.randint(-127, 128, size=(4096, 4096)).astype(np.int8)
    w_mat = np.random.randint(-127, 128, size=(4096, 4096)).astype(np.int8)
    scale_x = (0.02 * (1.0 + np.random.rand(1))).astype(np.float32)
    scale_w = (0.02 * (1.0 + np.random.rand(1))).astype(np.float32)
    return {"x": x, "w_mat": w_mat, "scale_x": scale_x, "scale_w": scale_w}

def reference(x, w_mat, scale_x, scale_w):
    acc = x.astype(np.float64) @ w_mat.astype(np.float64)
    y = (acc.astype(np.float32) * (scale_x * scale_w)).astype(np.float32)
    return (y / (1.0 + np.exp(-np.clip(y, -60.0, 60.0)))).astype(np.float32)
```

```cpp
#include <hip/hip_runtime.h>
#include <rccl/rccl.h>
#include <cstdio>
#include <cstdint>

#define HIP_CHECK(expr) do { hipError_t _e = (expr); if (_e != hipSuccess) { \
    fprintf(stderr, "HIP error %s at %s:%d\n", hipGetErrorString(_e), __FILE__, __LINE__); \
    return; } } while (0)

namespace {

#define GAS __attribute__((address_space(1)))
#define LAS __attribute__((address_space(3)))
typedef unsigned u32x4 __attribute__((ext_vector_type(4)));
typedef int i32x4 __attribute__((ext_vector_type(4)));
typedef float f32x4 __attribute__((ext_vector_type(4)));

constexpr int NR = 8;
constexpr int MS = 512;
constexpr int KS = 512;
constexpr int NN = 4096;
constexpr int NCH = 8;
constexpr int NWG = 256;
constexpr int NSEND = 64;
constexpr int NCW = 8;
constexpr int NTHREADS = (NCW + 1) * 64;
#define WIN 6

constexpr unsigned OFF_CNT = 0;
constexpr unsigned OFF_TMO = 256;
constexpr unsigned OFF_FLAGW = 4096;
constexpr unsigned OFF_FLAGX = 16384;
constexpr unsigned OFF_DATA = 1u << 20;
constexpr unsigned W_BYTES = 16u << 20;
constexpr unsigned X_BYTES = 2u << 20;
constexpr unsigned PAR_STRIDE = W_BYTES + X_BYTES;
constexpr size_t WS_NEEDED = (size_t)OFF_DATA + 2 * (size_t)PAR_STRIDE;

constexpr unsigned L_B = 0;
constexpr unsigned L_XS = 65536;
constexpr unsigned L_WS = 98304;
constexpr unsigned L_SYNC = 131072;
constexpr unsigned LDS_BYTES = L_SYNC + 256;

constexpr unsigned long long DEADLINE_TICKS = 100000000ull;

struct Params {
    const int8_t* x;
    const int8_t* w;
    const float* sx;
    const float* sw;
    float* out;
    char* ws[NR];
    int rank;
    int pad;
};

__device__ __forceinline__ void waitvm0() { asm volatile("s_waitcnt vmcnt(0)" ::: "memory"); }
__device__ __forceinline__ void waitwin() { asm volatile("s_waitcnt vmcnt(%0)" :: "n"(WIN) : "memory"); }
__device__ __forceinline__ void st16_wt(char* base, unsigned off, u32x4 v) {
    asm volatile("global_store_dwordx4 %0, %1, %2 sc0 sc1\n\ts_nop 1" :: "v"(off), "v"(v), "s"(base) : "memory");
}
__device__ __forceinline__ void lds_add(unsigned addr, unsigned v) {
    asm volatile("ds_add_u32 %0, %1" :: "v"(addr), "v"(v) : "memory");
}
__device__ __forceinline__ unsigned lds_ld(unsigned addr) {
    unsigned r;
    asm volatile("ds_read_b32 %0, %1\n\ts_waitcnt lgkmcnt(0)" : "=v"(r) : "v"(addr) : "memory");
    return r;
}
__device__ __forceinline__ void lds_wait_ge(unsigned addr, unsigned want) {
    while ((unsigned)__builtin_amdgcn_readfirstlane(lds_ld(addr)) < want) __builtin_amdgcn_s_sleep(1);
}
__device__ __forceinline__ char* uniform_ptr(char* p) {
    unsigned lo = (unsigned)__builtin_amdgcn_readfirstlane((unsigned)(size_t)p);
    unsigned hi = (unsigned)__builtin_amdgcn_readfirstlane((unsigned)((size_t)p >> 32));
    return (char*)(((size_t)hi << 32) | (size_t)lo);
}

__device__ __forceinline__ void wait_flags(GAS unsigned* flag, unsigned want, GAS unsigned* tmo, unsigned call) {
    unsigned long long t0 = __builtin_amdgcn_s_memrealtime();
    for (unsigned spins = 0;; ++spins) {
        unsigned v = __hip_atomic_load(flag, __ATOMIC_RELAXED, __HIP_MEMORY_SCOPE_SYSTEM);
        if (__all((int)(v - want) >= 0)) break;
        bool give_up = __builtin_amdgcn_s_memrealtime() - t0 > DEADLINE_TICKS;
        if ((spins & 15u) == 15u)
            give_up |= (unsigned)__builtin_amdgcn_readfirstlane(
                           __hip_atomic_load(tmo, __ATOMIC_RELAXED, __HIP_MEMORY_SCOPE_AGENT)) == call;
        if (give_up) {
            __hip_atomic_store(tmo, call, __ATOMIC_RELAXED, __HIP_MEMORY_SCOPE_AGENT);
            break;
        }
        __builtin_amdgcn_s_sleep(2);
    }
}

__global__ void __launch_bounds__(NTHREADS, 1) gemm_ag_kernel(Params p) {
    __shared__ __attribute__((aligned(16))) char smem[LDS_BYTES];
    const int tid = threadIdx.x;
    const int lane = tid & 63;
    const int wv = __builtin_amdgcn_readfirstlane(tid >> 6);
    const unsigned g = blockIdx.x;
    const unsigned lds0 = (unsigned)(size_t)(LAS char*)smem;
    const bool send_wg = g < NSEND;
    char* const myws = p.ws[0];
    GAS unsigned* const tmo = (GAS unsigned*)(myws + OFF_TMO);

    const unsigned snj = (g & 7u) * 8u + (g >> 3);

    if (wv == NCW) {
        if (!send_wg) return;
        const unsigned rb = g & 31u;
#pragma unroll
        for (int m = 0; m < 4; ++m) {
            const unsigned rel = (g >> 5) * 4u + ((m + g) & 3u);
            const unsigned d = (p.rank + rel) & 7u;
            const int8_t* src = p.x + (size_t)(d * MS + rb * 16u + (lane & 15)) * KS + (lane >> 4) * 16;
#pragma unroll
            for (int j = 0; j < NCH; ++j)
                __builtin_amdgcn_global_load_lds((const GAS void*)(src + j * 64),
                                                 (LAS void*)(smem + L_XS + (m * 8 + j) * 1024), 16, 0, 0);
        }
    } else {
        if (send_wg) {
#pragma unroll
            for (int t = 0; t < 4; ++t) {
                const unsigned row = (wv * 4 + t) * 16u + (lane >> 2);
                const int8_t* src = p.w + (size_t)row * NN + snj * 64u + (lane & 3) * 16;
                __builtin_amdgcn_global_load_lds((const GAS void*)src,
                                                 (LAS void*)(smem + L_WS + (wv * 4 + t) * 1024), 16, 0, 0);
            }
        }
        if (tid == 0) {
            GAS unsigned* cnt = (GAS unsigned*)(myws + OFF_CNT);
            unsigned c = __hip_atomic_load(cnt, __ATOMIC_RELAXED, __HIP_MEMORY_SCOPE_AGENT);
            unsigned call = c / NWG + 1u;
            LAS unsigned* s = (LAS unsigned*)(smem + L_SYNC);
            s[0] = call; s[1] = 0u; s[2] = 0u;
            asm volatile("" :: "v"(call) : "memory");
            (void)__hip_atomic_fetch_add(cnt, 1u, __ATOMIC_RELAXED, __HIP_MEMORY_SCOPE_AGENT);
            asm volatile("s_waitcnt lgkmcnt(0)" ::: "memory");
        }
    }
    __builtin_amdgcn_s_barrier();
    asm volatile("" ::: "memory");
    const unsigned call = (unsigned)__builtin_amdgcn_readfirstlane(lds_ld(lds0 + L_SYNC));
    const unsigned par = OFF_DATA + (call & 1u) * PAR_STRIDE;
    const unsigned fbase = (call - 1u) * NCH;

    if (wv == NCW) {
        const unsigned rb = g & 31u;
        char* peer = p.ws[0];
#pragma unroll
        for (int t = 1; t < NR; ++t) peer = ((lane & 7) == t) ? p.ws[t] : peer;
        char* dst[NR];
#pragma unroll
        for (int t = 0; t < NR; ++t) dst[t] = p.ws[(t + (g >> 3)) & 7u];
        char* xd[4];
#pragma unroll
        for (int m = 0; m < 4; ++m) xd[m] = p.ws[(g >> 5) * 4u + ((m + g) & 3u)];

        waitvm0();
        const unsigned xoff = par + W_BYTES + (p.rank * 32u + rb) * 8192u + lane * 16u;
#pragma unroll
        for (int m = 0; m < 4; ++m) {
#pragma unroll
            for (int j = 0; j < NCH; ++j) {
                u32x4 v = *(const LAS u32x4*)(smem + L_XS + (m * 8 + j) * 1024 + lane * 16);
                st16_wt(xd[m], xoff + j * 1024u, v);
                waitwin();
                if (m > 0 && j == WIN - 1) {
                    if (lane == 0)
                        __hip_atomic_store((GAS unsigned*)(xd[m - 1] + OFF_FLAGX + rb * 128u + p.rank * 4u), call,
                                           __ATOMIC_RELAXED, __HIP_MEMORY_SCOPE_SYSTEM);
                }
            }
        }
        lds_wait_ge(lds0 + L_SYNC + 8, NCW);
        GAS unsigned* fw = (GAS unsigned*)(peer + OFF_FLAGW + snj * 128u + p.rank * 4u);
        const unsigned c4 = lane & 15, kg = lane >> 4;
        for (int j = 0; j < NCH; ++j) {
            unsigned r[16];
#pragma unroll
            for (int t = 0; t < 16; ++t)
                r[t] = *(const LAS unsigned*)(smem + L_WS + (j * 64 + kg * 16 + t) * 64 + c4 * 4);
            u32x4 f[4];
#pragma unroll
            for (int q = 0; q < 4; ++q) {
                unsigned t0 = __builtin_amdgcn_perm(r[4 * q + 1], r[4 * q + 0], 0x05010400u);
                unsigned t1 = __builtin_amdgcn_perm(r[4 * q + 1], r[4 * q + 0], 0x07030602u);
                unsigned t2 = __builtin_amdgcn_perm(r[4 * q + 3], r[4 * q + 2], 0x05010400u);
                unsigned t3 = __builtin_amdgcn_perm(r[4 * q + 3], r[4 * q + 2], 0x07030602u);
                f[0][q] = __builtin_amdgcn_perm(t2, t0, 0x05040100u);
                f[1][q] = __builtin_amdgcn_perm(t2, t0, 0x07060302u);
                f[2][q] = __builtin_amdgcn_perm(t3, t1, 0x05040100u);
                f[3][q] = __builtin_amdgcn_perm(t3, t1, 0x07060302u);
            }
            const unsigned woff = par + ((p.rank * NCH + j) * 64u + snj) * 4096u + lane * 16u;
#pragma unroll
            for (int i = 0; i < 4; ++i) {
#pragma unroll
                for (int t = 0; t < NR; ++t) {
                    st16_wt(dst[t], woff + i * 1024u, f[i]);
                    waitwin();
                    if (i * NR + t == WIN - 1) {
                        if (j == 0) {
                            if (lane == 0)
                                __hip_atomic_store((GAS unsigned*)(xd[3] + OFF_FLAGX + rb * 128u + p.rank * 4u), call,
                                                   __ATOMIC_RELAXED, __HIP_MEMORY_SCOPE_SYSTEM);
                        } else if (lane < NR) {
                            __hip_atomic_store(fw, fbase + j, __ATOMIC_RELAXED, __HIP_MEMORY_SCOPE_SYSTEM);
                        }
                    }
                }
            }
        }
        waitvm0();
        if (lane < NR) __hip_atomic_store(fw, fbase + NCH, __ATOMIC_RELAXED, __HIP_MEMORY_SCOPE_SYSTEM);
        return;
    }

    if (send_wg) {
        waitvm0();
        if (lane == 0) lds_add(lds0 + L_SYNC + 8, 1u);
    }
    const unsigned nj = (g & 7u) * 8u + (g >> 5);
    const unsigned mi = (g >> 3) & 3u;
    const unsigned rb = mi * 8u + wv;
    const __amdgpu_buffer_rsrc_t rsX =
        __builtin_amdgcn_make_buffer_rsrc((void*)(myws + par + W_BYTES), 0, (int)X_BYTES, 0x00020000);
    const char* wsrc = myws + par + ((unsigned)wv * NCH * 64u + nj) * 4096u + lane * 16;
    GAS unsigned* fw = (GAS unsigned*)(myws + OFF_FLAGW + nj * 128u + wv * 4u);

    wait_flags((GAS unsigned*)(myws + OFF_FLAGX + rb * 128u + (lane & 7) * 4u), call, tmo, call);

    i32x4 acc[4] = {};
    for (int j = 0; j < NCH; ++j) {
        u32x4 a[NR];
#pragma unroll
        for (int q = 0; q < NR; ++q)
            a[q] = __builtin_amdgcn_raw_buffer_load_b128(rsX, (int)(((q * 32u + rb) * 8u + j) * 1024u + lane * 16u), 0, 17);
        wait_flags(fw, fbase + j + 1u, tmo, call);
        char* lb = smem + L_B + (j & 1) * 32768;
#pragma unroll
        for (int i = 0; i < 4; ++i)
            __builtin_amdgcn_global_load_lds((const GAS void*)(wsrc + (size_t)j * (64u * 4096u) + i * 1024),
                                             (LAS void*)(lb + wv * 4096 + i * 1024), 16, 0, 17);
        waitvm0();
        if (lane == 0) lds_add(lds0 + L_SYNC + 4, 1u);
        lds_wait_ge(lds0 + L_SYNC + 4, NCW * (j + 1));
#pragma unroll
        for (int q = 0; q < NR; ++q) {
#pragma unroll
            for (int i = 0; i < 4; ++i) {
                i32x4 b = *(const LAS i32x4*)(lb + q * 4096 + i * 1024 + lane * 16);
                acc[i] = __builtin_amdgcn_mfma_i32_16x16x64_i8(__builtin_bit_cast(i32x4, a[q]), b, acc[i], 0, 0, 0);
            }
        }
    }

    const bool bad = (unsigned)__builtin_amdgcn_readfirstlane(
                         __hip_atomic_load(tmo, __ATOMIC_RELAXED, __HIP_MEMORY_SCOPE_AGENT)) == call;
    const float scale = p.sx[0] * p.sw[0];
    float* orow = p.out + (size_t)(rb * 16u + (lane >> 4) * 4u) * NN + nj * 64u + (lane & 15) * 4u;
#pragma unroll
    for (int r = 0; r < 4; ++r) {
        f32x4 o;
#pragma unroll
        for (int i = 0; i < 4; ++i) {
            float y = (float)acc[i][r] * scale;
            float yc = __builtin_amdgcn_fmed3f(y, -60.0f, 60.0f);
            float e = __builtin_amdgcn_exp2f(yc * -1.4426950408889634f);
            o[i] = y * __builtin_amdgcn_rcpf(1.0f + e);
        }
        if (bad) o = (f32x4){__builtin_nanf(""), __builtin_nanf(""), __builtin_nanf(""), __builtin_nanf("")};
        *(f32x4*)(orow + (size_t)r * NN) = o;
    }
}

}

extern "C" void kernel_launch(int rank, int nranks, ncclComm_t comm,
                              hipStream_t stream,
                              void* const* d_in, const int* in_sizes, int n_in,
                              void* d_out, int out_size,
                              const int* in_dtypes, int out_dtype,
                              void* const* d_out_all,
                              void* const* d_ws_all, size_t ws_bytes) {
    (void)comm; (void)in_dtypes; (void)out_dtype; (void)d_out_all;
    if (nranks != NR || (unsigned)rank >= (unsigned)NR || n_in != 4 || ws_bytes < WS_NEEDED ||
        in_sizes[0] != 4096 * KS || in_sizes[1] != KS * NN || out_size != MS * NN) {
        fprintf(stderr, "kernel_launch: unexpected problem shape\n");
        return;
    }
    Params p;
    p.x = (const int8_t*)d_in[0];
    p.w = (const int8_t*)d_in[1];
    p.sx = (const float*)d_in[2];
    p.sw = (const float*)d_in[3];
    p.out = (float*)d_out;
    for (int t = 0; t < NR; ++t) p.ws[t] = (char*)d_ws_all[(rank + t) % NR];
    p.rank = rank;
    p.pad = 0;
    hipLaunchKernelGGL(gemm_ag_kernel, dim3(NWG), dim3(NTHREADS), 0, stream, p);
    HIP_CHECK(hipGetLastError());
}
```

```cpp
#include <hip/hip_runtime.h>
#include <rccl/rccl.h>
#include <cstdio>
#include <cstdint>

#define HIP_CHECK(expr) do { hipError_t _e = (expr); if (_e != hipSuccess) { \
    fprintf(stderr, "HIP error %s at %s:%d\n", hipGetErrorString(_e), __FILE__, __LINE__); \
    return; } } while (0)

namespace {

#define GAS __attribute__((address_space(1)))
#define LAS __attribute__((address_space(3)))
typedef unsigned u32x4 __attribute__((ext_vector_type(4)));
typedef int i32x4 __attribute__((ext_vector_type(4)));
typedef float f32x4 __attribute__((ext_vector_type(4)));

constexpr int NR = 8;
constexpr int MS = 512;
constexpr int KS = 512;
constexpr int NN = 4096;
constexpr int NCH = 8;
constexpr int NWG = 256;
constexpr int NUNIT = NCH * 64;
constexpr int NCW = 8;
constexpr int W_LOAD = 8, W_SEND = 9;
constexpr int NWAVE = 10;
constexpr int NTHREADS = NWAVE * 64;
#define NSEND 32
#define WIN 14
#define LD0 12
#define LDN (WIN > 14 ? -1 : 12)
#define XWIN 1
#define QLAST 0
constexpr unsigned NFULL = QLAST ? (NCH - 1) * 64 : NCH * 64;
constexpr unsigned NJOB = NFULL + (QLAST ? 64 * 4 : 0);
static_assert(WIN >= 2 && WIN <= 28 && WIN + 12 <= 63 && LD0 >= 2 && LD0 <= 20 && NSEND <= 64, "sender window");
constexpr unsigned TICKETS = NJOB;

constexpr unsigned OFF_CNT = 0;
constexpr unsigned OFF_TMO = 256;
constexpr unsigned OFF_HEAD = 512;
constexpr unsigned OFF_FLAGT = 4096;
constexpr unsigned OFF_FLAGX = 8192;
constexpr unsigned OFF_FLAGW = 16384;
constexpr unsigned OFF_FLAGQ = OFF_FLAGW + NFULL * 8 * 4;
constexpr unsigned OFF_DATA = 1u << 20;
constexpr unsigned W_BYTES = 16u << 20;
constexpr unsigned X_BYTES = 2u << 20;
constexpr unsigned PAR_STRIDE = W_BYTES + X_BYTES;
constexpr size_t WS_NEEDED = (size_t)OFF_DATA + 2 * (size_t)PAR_STRIDE;

constexpr unsigned L_B = 0;
constexpr unsigned L_RING = 98304;
constexpr unsigned L_SEQ = 106496;
constexpr unsigned L_SYNC = 106752;
constexpr unsigned S_CALL = L_SYNC, S_RING = L_SYNC + 8, S_TDONE = L_SYNC + 12, S_BAD = L_SYNC + 16;
constexpr unsigned S_DONE = L_SYNC + 32;
constexpr unsigned LDS_BYTES = L_SYNC + 256;

constexpr unsigned long long DEADLINE_TICKS = 100000000ull;

struct Params {
    const int8_t* x;
    const int8_t* w;
    const float* sx;
    const float* sw;
    float* out;
    char* ws[NR];
    int rank;
    int pad;
};

#define STAMP(i) do {} while (0)
#define STAMP_DUMP() do {} while (0)
constexpr unsigned LDS_TOTAL = LDS_BYTES;

template <int N> __device__ __forceinline__ void waitvm() { asm volatile("s_waitcnt vmcnt(%0)" :: "n"(N) : "memory"); }
template <int N> __device__ __forceinline__ void waitvm_n(int n) {
    if (n == 0) waitvm<N>(); else if (n == 1) waitvm<N + 1>(); else if (n == 2) waitvm<N + 2>(); else if (n == 3) waitvm<N + 3>();
    else if (n == 4) waitvm<N + 4>(); else if (n == 5) waitvm<N + 5>(); else if (n == 6) waitvm<N + 6>();
    else if (n == 7) waitvm<N + 7>(); else if (n == 8) waitvm<N + 8>(); else if (n == 9) waitvm<N + 9>();
    else if (n == 10) waitvm<N + 10>(); else if (n == 11) waitvm<N + 11>(); else waitvm<N + 12>();
}
__device__ __forceinline__ void st16_wt(char* base, unsigned off, u32x4 v) {
    asm volatile("global_store_dwordx4 %0, %1, %2 sc0 sc1\n\ts_nop 1" :: "v"(off), "v"(v), "s"(base) : "memory");
}
__device__ __forceinline__ void glds16_sys(const void* gsrc, unsigned lds_dst) {
    unsigned keep;
    asm volatile("s_mov_b32 %0, m0\n\ts_mov_b32 m0, %2\n\ts_nop 0\n\tglobal_load_lds_dwordx4 %1, off sc0 sc1\n\ts_mov_b32 m0, %0"
                 : "=&s"(keep) : "v"(gsrc), "s"(lds_dst) : "memory");
}
__device__ __forceinline__ void glds64_sys(const void* gsrc, unsigned lds_dst) {
    unsigned keep;
    asm volatile("s_mov_b32 %0, m0\n\ts_mov_b32 m0, %2\n\ts_nop 0\n\t"
                 "global_load_lds_dwordx4 %1, off sc0 sc1\n\tglobal_load_lds_dwordx4 %1, off offset:1024 sc0 sc1\n\t"
                 "global_load_lds_dwordx4 %1, off offset:2048 sc0 sc1\n\tglobal_load_lds_dwordx4 %1, off offset:3072 sc0 sc1\n\t"
                 "s_mov_b32 m0, %0"
                 : "=&s"(keep) : "v"(gsrc), "s"(lds_dst) : "memory");
}
__device__ __forceinline__ void lds_rd4(u32x4 (&f)[4], unsigned addr) {
    asm volatile("ds_read_b128 %0, %4\n\tds_read_b128 %1, %4 offset:1024\n\tds_read_b128 %2, %4 offset:2048\n\t"
                 "ds_read_b128 %3, %4 offset:3072\n\ts_waitcnt lgkmcnt(0)"
                 : "=&v"(f[0]), "=&v"(f[1]), "=&v"(f[2]), "=&v"(f[3]) : "v"(addr) : "memory");
}
__device__ __forceinline__ u32x4 lds_rd1(unsigned addr) {
    u32x4 f;
    asm volatile("ds_read_b128 %0, %1\n\ts_waitcnt lgkmcnt(0)" : "=v"(f) : "v"(addr) : "memory");
    return f;
}
__device__ __forceinline__ void ticket_issue(unsigned& q, GAS unsigned* head) {
    unsigned zero = 0u, one = 1u;
    asm volatile("global_atomic_add %0, %1, %2, %3 sc0" : "=v"(q) : "v"(zero), "v"(one), "s"(head) : "memory");
}
__device__ __forceinline__ void lds_add(unsigned addr, unsigned v) {
    asm volatile("ds_add_u32 %0, %1" :: "v"(addr), "v"(v) : "memory");
}
__device__ __forceinline__ void lds_st(unsigned addr, unsigned v) {
    asm volatile("ds_write_b32 %0, %1" :: "v"(addr), "v"(v) : "memory");
}
__device__ __forceinline__ unsigned lds_ld(unsigned addr) {
    unsigned r;
    asm volatile("ds_read_b32 %0, %1\n\ts_waitcnt lgkmcnt(0)" : "=v"(r) : "v"(addr) : "memory");
    return r;
}
__device__ __forceinline__ unsigned lds_ld_u(unsigned addr) { return (unsigned)__builtin_amdgcn_readfirstlane(lds_ld(addr)); }
__device__ __forceinline__ void lds_wait_ge(unsigned addr, unsigned want) {
    while (lds_ld_u(addr) < want) __builtin_amdgcn_s_sleep(1);
}

__device__ __forceinline__ bool give_up(unsigned long long t0, unsigned spins, GAS unsigned* tmo, unsigned call) {
    bool g = __builtin_amdgcn_s_memrealtime() - t0 > DEADLINE_TICKS;
    if ((spins & 15u) == 15u)
        g |= (unsigned)__builtin_amdgcn_readfirstlane(
                 __hip_atomic_load(tmo, __ATOMIC_RELAXED, __HIP_MEMORY_SCOPE_AGENT)) == call;
    if (g) __hip_atomic_store(tmo, call, __ATOMIC_RELAXED, __HIP_MEMORY_SCOPE_AGENT);
    return g;
}
__device__ __forceinline__ void wait_flags(GAS unsigned* flag, unsigned want, GAS unsigned* tmo, unsigned call) {
    unsigned long long t0 = __builtin_amdgcn_s_memrealtime();
    for (unsigned spins = 0;; ++spins) {
        unsigned v = __hip_atomic_load(flag, __ATOMIC_RELAXED, __HIP_MEMORY_SCOPE_SYSTEM);
        if (__all((int)(v - want) >= 0)) break;
        if (give_up(t0, spins, tmo, call)) break;
        __builtin_amdgcn_s_sleep(2);
    }
}

template <int LDP, bool FIRST>
__device__ __forceinline__ void send_unit(char* const (&dst)[NR - 1], unsigned woff, const u32x4 (&f)[4], const char* nsrc,
                                          unsigned nslot, unsigned& qn, GAS unsigned* head, GAS unsigned* fprev, bool fact,
                                          unsigned call, unsigned tdone_addr, int lane) {
    if (LDP < 0) {
        if (lane == 0) ticket_issue(qn, head);
        glds64_sys(nsrc, nslot);
    }
#pragma unroll
    for (int k = 0; k < 28; ++k) {
        st16_wt(dst[k % 7], woff + (k / 7) * 1024u, f[k / 7]);
        constexpr int pos[6] = {WIN - 1, LDP, LDP, LDP, LDP, LDP};
        int n = 0;
#pragma unroll
        for (int e = 0; e < 6; ++e)
            n += ((pos[e] >= k - WIN && pos[e] <= k - 1) ? 1 : 0) + ((pos[e] - 28 >= k - WIN) ? 1 : 0);
        waitvm_n<WIN>(n);
        if (k == WIN - 1) {
            waitvm<WIN>();
            if (fact) __hip_atomic_store(fprev, call, __ATOMIC_RELAXED, __HIP_MEMORY_SCOPE_SYSTEM);
        }
        if (k == LDP) {
            if (FIRST) lds_wait_ge(tdone_addr, 1u);
            if (lane == 0) ticket_issue(qn, head);
            glds64_sys(nsrc, nslot);
        }
    }
    constexpr int younger = (27 - LDP) + (WIN - 1 > LDP ? 1 : 0);
    waitvm<(younger < 63 ? younger : 63)>();
    asm volatile("" : "+v"(qn));
}

__global__ void __launch_bounds__(NTHREADS, 1) gemm_ag_kernel(Params p) {
    __shared__ __attribute__((aligned(16))) char smem[LDS_TOTAL];
    const int tid = threadIdx.x;
    const int lane = tid & 63;
    const int wv = __builtin_amdgcn_readfirstlane(tid >> 6);
    const unsigned g = blockIdx.x;
    const unsigned lds0 = (unsigned)(size_t)(LAS char*)smem;
    const bool send_wg = g < NSEND;
    char* const myws = p.ws[0];
    GAS unsigned* const tmo = (GAS unsigned*)(myws + OFF_TMO);
    STAMP(0);

    const unsigned xrb = g >> 3;
    const unsigned xrel = (g + (g >> 3)) & 7u;
    u32x4 xv[NCH];
    unsigned tr[16];
    const unsigned tu = ((g + 256u * wv) & ~63u) | ((g + 8u * p.rank) & 63u);
    unsigned q0 = 0;
    GAS unsigned* const head = (GAS unsigned*)(myws + OFF_HEAD);

    if (wv == W_SEND) {
        const __amdgpu_buffer_rsrc_t rs = __builtin_amdgcn_make_buffer_rsrc((void*)p.x, 0, 4096 * KS, 0x00020000);
        const unsigned d = (p.rank + xrel) & 7u;
#pragma unroll
        for (int j = 0; j < NCH; ++j)
            xv[j] = __builtin_amdgcn_raw_buffer_load_b128(
                rs, (int)((d * MS + xrb * 16u + (lane & 15)) * KS + j * 64u + (lane >> 4) * 16u), 0, 0);
        if (send_wg && lane == 0) ticket_issue(q0, head);
    } else if (wv == W_LOAD) {
        unsigned c = 0;
        if (lane == 0) c = __hip_atomic_load((GAS unsigned*)(myws + OFF_CNT), __ATOMIC_RELAXED, __HIP_MEMORY_SCOPE_AGENT);
        if (lane < NCH) *(LAS unsigned*)(smem + L_SEQ + lane * 4) = 0u;
        if (lane >= 1 && lane < 16) *(LAS unsigned*)(smem + L_SYNC + lane * 4) = 0u;
        if (lane == 0) *(LAS unsigned*)(smem + S_CALL) = c / NWG + 1u;
        asm volatile("s_waitcnt lgkmcnt(0)" ::: "memory");
    } else if (wv < 2) {
        const unsigned u = tu;
        const __amdgpu_buffer_rsrc_t rs = __builtin_amdgcn_make_buffer_rsrc((void*)p.w, 0, KS * NN, 0x00020000);
#pragma unroll
        for (int t = 0; t < 16; ++t)
            tr[t] = __builtin_amdgcn_raw_buffer_load_b32(
                rs, (int)(((u >> 6) * 64u + (lane >> 4) * 16u + t) * NN + (u & 63u) * 64u + (lane & 15) * 4u), 0, 0);
    }
    __builtin_amdgcn_s_barrier();
    asm volatile("" ::: "memory");
    const unsigned call = lds_ld_u(lds0 + S_CALL);
    const unsigned par = OFF_DATA + (call & 1u) * PAR_STRIDE;
    const unsigned nj = (g & 7u) * 8u + (g >> 5);
    STAMP(1);

    if (wv == W_SEND) {
        __builtin_amdgcn_s_setprio(2);
        char* const xd = p.ws[xrel];
        const unsigned xoff = par + W_BYTES + (p.rank * 32u + xrb) * 8192u + lane * 16u;
        waitvm<0>();
#pragma unroll
        for (int j = 0; j < NCH; ++j) asm volatile("" : "+v"(xv[j]));
        asm volatile("" : "+v"(q0));
        STAMP(2);
#pragma unroll
        for (int j = 0; j < NCH; ++j) {
            st16_wt(xd, xoff + j * 1024u, xv[j]);
            if (j < NCH - 1) waitvm<XWIN>();
        }
        GAS unsigned* const fx = (GAS unsigned*)(xd + OFF_FLAGX + xrb * 128u + p.rank * 4u);
        if (!send_wg) {
            waitvm<0>();
            if (lane == 0) __hip_atomic_store(fx, call, __ATOMIC_RELAXED, __HIP_MEMORY_SCOPE_SYSTEM);
            STAMP(3);
            STAMP_DUMP();
            return;
        }
        STAMP(3);
        char* peer = p.ws[0];
#pragma unroll
        for (int t = 1; t < NR; ++t) peer = ((lane & 7) == t) ? p.ws[t] : peer;
        char* dst[NR - 1];
#pragma unroll
        for (int t = 0; t < NR - 1; ++t) dst[t] = p.ws[1 + (t + g) % 7u];
        const unsigned qbase = (call - 1u) * TICKETS - (unsigned)NSEND;
        unsigned jcur = g;
        unsigned jnxt = (unsigned)__builtin_amdgcn_readfirstlane(q0) - qbase;
        const unsigned ring = (unsigned)__builtin_amdgcn_readfirstlane(lds0 + L_RING);
        const unsigned wbase = par + p.rank * (NUNIT * 4096u);
        const char* wloc = myws + wbase + lane * 16;
        GAS unsigned* fprev = fx;
        bool fact = lane == 0;
        lds_wait_ge(lds0 + S_RING, 1u);
        unsigned slot = 0;
        bool first = true;
#define ROT(u) (((u) & ~63u) | (((u) + 8u * p.rank) & 63u))
#define JOB_UNIT(jb) ROT((jb) < NFULL ? (jb) : NFULL + (((jb) - NFULL) >> 2))
#define JOB_FLAG(jb) ((GAS unsigned*)(peer + ((jb) < NFULL ? OFF_FLAGW + (JOB_UNIT(jb) * 8u + p.rank) * 4u \
                                                          : OFF_FLAGQ + (((JOB_UNIT(jb) - NFULL) * 8u + p.rank) * 4u + (((jb) - NFULL) & 3u)) * 4u)))
        while (jnxt < NJOB) {
            const char* nsrc = wloc + JOB_UNIT(jnxt) * 4096u;
            const unsigned nslot = ring + (slot ^ 1u) * 4096u;
            unsigned qn;
            if (jcur < NFULL) {
                u32x4 f[4];
                lds_rd4(f, ring + slot * 4096u + lane * 16u);
                const unsigned woff = wbase + JOB_UNIT(jcur) * 4096u + lane * 16u;
                STAMP(4 + (jcur >> 6));
                if (first) send_unit<LD0, true>(dst, woff, f, nsrc, nslot, qn, head, fprev, fact, call, lds0 + S_TDONE, lane);
                else send_unit<LDN, false>(dst, woff, f, nsrc, nslot, qn, head, fprev, fact, call, 0u, lane);
            } else {
                const unsigned fr = (jcur - NFULL) & 3u;
                u32x4 f = lds_rd1(ring + slot * 4096u + fr * 1024u + lane * 16u);
                const unsigned woff = wbase + JOB_UNIT(jcur) * 4096u + fr * 1024u + lane * 16u;
                STAMP(11);
                if (lane == 0) ticket_issue(qn, head);
                glds64_sys(nsrc, nslot);
#pragma unroll
                for (int k = 0; k < 7; ++k) st16_wt(dst[k], woff, f);
                waitvm<7>();
                if (fact) __hip_atomic_store(fprev, call, __ATOMIC_RELAXED, __HIP_MEMORY_SCOPE_SYSTEM);
            }
            asm volatile("" : "+v"(qn));
            fprev = JOB_FLAG(jcur);
            fact = lane >= 1 && lane < NR;
            first = false;
            jcur = jnxt;
            jnxt = (unsigned)__builtin_amdgcn_readfirstlane(qn) - qbase;
            slot ^= 1u;
        }
        if (jcur < NFULL) {
            u32x4 f[4];
            lds_rd4(f, ring + slot * 4096u + lane * 16u);
            const unsigned woff = wbase + JOB_UNIT(jcur) * 4096u + lane * 16u;
#pragma unroll
            for (int k = 0; k < 28; ++k) {
                st16_wt(dst[k % 7], woff + (k / 7) * 1024u, f[k / 7]);
                if (k == WIN - 1) {
                    waitvm<WIN>();
                    if (fact) __hip_atomic_store(fprev, call, __ATOMIC_RELAXED, __HIP_MEMORY_SCOPE_SYSTEM);
                }
            }
        } else {
            const unsigned fr = (jcur - NFULL) & 3u;
            u32x4 f = lds_rd1(ring + slot * 4096u + fr * 1024u + lane * 16u);
            const unsigned woff = wbase + JOB_UNIT(jcur) * 4096u + fr * 1024u + lane * 16u;
#pragma unroll
            for (int k = 0; k < 7; ++k) st16_wt(dst[k], woff, f);
            waitvm<7>();
            if (fact) __hip_atomic_store(fprev, call, __ATOMIC_RELAXED, __HIP_MEMORY_SCOPE_SYSTEM);
        }
        fprev = JOB_FLAG(jcur);
        STAMP(12);
        waitvm<0>();
        STAMP(13);
        if (lane >= 1 && lane < NR) __hip_atomic_store(fprev, call, __ATOMIC_RELAXED, __HIP_MEMORY_SCOPE_SYSTEM);
        STAMP_DUMP();
        return;
    }

    if (wv == W_LOAD) {
        __builtin_amdgcn_s_setprio(3);
        if (send_wg) {
            const char* ft = myws + OFF_FLAGT + lane * 16;
            unsigned long long t0 = __builtin_amdgcn_s_memrealtime();
            for (unsigned spins = 0;; ++spins) {
                u32x4 a, b;
                asm volatile("global_load_dwordx4 %0, %2, off sc0 sc1\n\tglobal_load_dwordx4 %1, %2, off offset:1024 sc0 sc1\n\t"
                             "s_waitcnt vmcnt(0)" : "=&v"(a), "=&v"(b) : "v"(ft) : "memory");
                bool ok = true;
#pragma unroll
                for (int e = 0; e < 4; ++e) ok = ok && (int)(a[e] - call) >= 0 && (int)(b[e] - call) >= 0;
                if (__all(ok)) break;
                if (give_up(t0, spins, tmo, call)) break;
                __builtin_amdgcn_s_sleep(2);
            }
            if (lane == 0) lds_st(lds0 + S_TDONE, 1u);
            STAMP(12);
        }
        const char* wsrc = myws + par + nj * 4096u + lane * 16;
        const unsigned lbase = (unsigned)__builtin_amdgcn_readfirstlane(lds0 + L_B);
        unsigned cs0 = 0, cs1 = 1;
        bool act0 = false, act1 = false;
        unsigned pub0 = 0, pub1 = 0, tmo_now = 0;
        unsigned long long pending = 0, issued = 0;
        bool counted = false;
        const unsigned long long t0 = __builtin_amdgcn_s_memrealtime();
        for (unsigned spins = 0;; ++spins) {
            if (!act0 && cs0 < NCH && (cs0 < 3 || lds_ld_u(lds0 + S_DONE + (cs0 - 3) * 4u) >= NCW)) {
                act0 = true; pub0 = 0; pending |= 0xffull; STAMP(16 + cs0);
            }
            if (!act1 && cs1 < NCH && (cs1 < 3 || lds_ld_u(lds0 + S_DONE + (cs1 - 3) * 4u) >= NCW)) {
                act1 = true; pub1 = 0; pending |= (QLAST && cs1 == NCH - 1) ? 0xffffffff00000000ull : 0xff00ull; STAMP(16 + cs1);
            }
            unsigned long long ready = 0;
            if (pending) {
                const unsigned cl = (lane & 8) ? cs1 : cs0;
                const unsigned fo = lane >= 32 ? OFF_FLAGQ + nj * 128u + (lane - 32) * 4u
                                               : OFF_FLAGW + (((cl < NFULL / 64 ? cl : 0u) * 64u + nj) * 8u + (lane & 7)) * 4u;
                unsigned v = __hip_atomic_load((GAS unsigned*)(myws + fo), __ATOMIC_RELAXED, __HIP_MEMORY_SCOPE_SYSTEM);
                ready = __ballot((int)(v - call) >= 0) & pending;
                if (!ready && give_up(t0, spins, tmo, call)) ready = pending;
            }
            waitvm<0>();
            unsigned n0 = pub0, n1 = pub1;
            if (act0) n0 = (unsigned)issued & 0xffu;
            if (act1) {
                if (QLAST && cs1 == NCH - 1) {
                    const unsigned hi = (unsigned)(issued >> 32);
                    const unsigned f4 = hi & (hi >> 1) & (hi >> 2) & (hi >> 3);
                    n1 = 0;
#pragma unroll
                    for (int q = 0; q < NR; ++q) n1 |= ((f4 >> (4 * q)) & 1u) << q;
                } else {
                    n1 = (unsigned)(issued >> 8) & 0xffu;
                }
            }
            if (!pending && cs0 >= NCH - 2 && cs1 >= NCH - 1 && (act0 || cs0 >= NCH) && (act1 || cs1 >= NCH) && lane == 0)
                lds_st(lds0 + S_BAD, tmo_now == call ? 1u : 0u);
            if (act0 && n0 != pub0) {
                pub0 = n0;
                if (lane == 0) lds_st(lds0 + L_SEQ + cs0 * 4u, pub0);
                if (pub0 == 0xffu) { STAMP(4 + cs0); act0 = false; cs0 += 2; issued &= ~0xffull; }
            }
            if (act1 && n1 != pub1) {
                pub1 = n1;
                if (lane == 0) lds_st(lds0 + L_SEQ + cs1 * 4u, pub1);
                if (pub1 == 0xffu) { STAMP(4 + cs1); act1 = false; cs1 += 2; issued &= 0xffull; }
            }
            if (cs0 >= NCH && cs1 >= NCH) break;
            if (!counted && cs0 >= 2) {
                counted = true;
                if (lane == 0)
                    (void)__hip_atomic_fetch_add((GAS unsigned*)(myws + OFF_CNT), 1u, __ATOMIC_RELAXED, __HIP_MEMORY_SCOPE_AGENT);
            }
            if (!ready) { __builtin_amdgcn_s_sleep(1); continue; }
            pending &= ~ready;
            issued |= ready;
            while (ready) {
                const unsigned b = __builtin_ctzll(ready);
                ready &= ready - 1;
                if (b >= 32) {
                    const unsigned q = (b - 32) >> 2, i = b & 3u;
                    glds16_sys(wsrc + (q * NCH + (NCH - 1)) * (64u * 4096u) + i * 1024u,
                               lbase + ((NCH - 1) % 3) * 32768u + q * 4096u + i * 1024u);
                } else {
                    const unsigned c = b >= 8 ? cs1 : cs0, q = b & 7u;
                    glds64_sys(wsrc + (q * NCH + c) * (64u * 4096u), lbase + (c % 3u) * 32768u + q * 4096u);
                }
            }
            if (!pending && cs0 >= NCH - 2 && cs1 >= NCH - 1)
                tmo_now = (unsigned)__builtin_amdgcn_readfirstlane(__hip_atomic_load(tmo, __ATOMIC_RELAXED, __HIP_MEMORY_SCOPE_AGENT));
        }
        STAMP_DUMP();
        return;
    }

    if (wv < 2) {
        const unsigned u = tu;
        u32x4 f[4];
#pragma unroll
        for (int q = 0; q < 4; ++q) {
            unsigned t0 = __builtin_amdgcn_perm(tr[4 * q + 1], tr[4 * q + 0], 0x05010400u);
            unsigned t1 = __builtin_amdgcn_perm(tr[4 * q + 1], tr[4 * q + 0], 0x07030602u);
            unsigned t2 = __builtin_amdgcn_perm(tr[4 * q + 3], tr[4 * q + 2], 0x05010400u);
            unsigned t3 = __builtin_amdgcn_perm(tr[4 * q + 3], tr[4 * q + 2], 0x07030602u);
            f[0][q] = __builtin_amdgcn_perm(t2, t0, 0x05040100u);
            f[1][q] = __builtin_amdgcn_perm(t2, t0, 0x07060302u);
            f[2][q] = __builtin_amdgcn_perm(t3, t1, 0x05040100u);
            f[3][q] = __builtin_amdgcn_perm(t3, t1, 0x07060302u);
        }
        const unsigned woff = par + (p.rank * NUNIT + u) * 4096u + lane * 16u;
#pragma unroll
        for (int i = 0; i < 4; ++i) st16_wt(myws, woff + i * 1024u, f[i]);
        if (send_wg && wv == 0) {
#pragma unroll
            for (int i = 0; i < 4; ++i) *(LAS u32x4*)(smem + L_RING + i * 1024 + lane * 16) = f[i];
            asm volatile("s_waitcnt lgkmcnt(0)" ::: "memory");
            if (lane == 0) lds_st(lds0 + S_RING, 1u);
        }
        waitvm<0>();
        if (lane == 0) __hip_atomic_store((GAS unsigned*)(myws + OFF_FLAGT + u * 4u), call, __ATOMIC_RELAXED, __HIP_MEMORY_SCOPE_SYSTEM);
        if (u < NFULL) {
            if (lane == 0)
                __hip_atomic_store((GAS unsigned*)(myws + OFF_FLAGW + (u * 8u + p.rank) * 4u), call, __ATOMIC_RELAXED, __HIP_MEMORY_SCOPE_SYSTEM);
        } else if (lane < 4) {
            __hip_atomic_store((GAS unsigned*)(myws + OFF_FLAGQ + (((u - NFULL) * 8u + p.rank) * 4u + lane) * 4u), call,
                               __ATOMIC_RELAXED, __HIP_MEMORY_SCOPE_SYSTEM);
        }
    }
    const float scale = p.sx[0] * p.sw[0];
    const unsigned mi = (g >> 3) & 3u;
    const unsigned rb = mi * 8u + wv;
    const __amdgpu_buffer_rsrc_t rsX =
        __builtin_amdgcn_make_buffer_rsrc((void*)(myws + par + W_BYTES), 0, (int)X_BYTES, 0x00020000);

    wait_flags((GAS unsigned*)(myws + OFF_FLAGX + rb * 128u + (lane & 7) * 4u), call, tmo, call);

    STAMP(2);
    i32x4 acc[4] = {};
#pragma unroll 1
    for (int j = 0; j < NCH; ++j) {
        u32x4 a[NR];
#pragma unroll
        for (int q = 0; q < NR; ++q)
            a[q] = __builtin_amdgcn_raw_buffer_load_b128(rsX, (int)(((q * 32u + rb) * 8u + j) * 1024u + lane * 16u), 0, 17);
        const char* lb = smem + L_B + (j % 3) * 32768 + lane * 16;
        unsigned donem = 0;
        while (donem != 0xffu) {
            unsigned m = lds_ld_u(lds0 + L_SEQ + j * 4u) & ~donem;
            if (!m) { __builtin_amdgcn_s_sleep(1); continue; }
            if (donem == 0) STAMP(4 + j * 3);
            donem |= m;
            do {
                const unsigned q = __builtin_ctz(m);
                m &= m - 1;
                i32x4 b[4];
#pragma unroll
                for (int i = 0; i < 4; ++i) b[i] = *(const LAS i32x4*)(lb + q * 4096u + i * 1024);
                u32x4 av;
                switch (q) {
                    case 0: av = a[0]; break; case 1: av = a[1]; break; case 2: av = a[2]; break; case 3: av = a[3]; break;
                    case 4: av = a[4]; break; case 5: av = a[5]; break; case 6: av = a[6]; break; default: av = a[7]; break;
                }
#pragma unroll
                for (int i = 0; i < 4; ++i)
                    acc[i] = __builtin_amdgcn_mfma_i32_16x16x64_i8(__builtin_bit_cast(i32x4, av), b[i], acc[i], 0, 0, 0);
            } while (m);
        }
        STAMP(5 + j * 3);
        if (lane == 0) lds_add(lds0 + S_DONE + j * 4u, 1u);
    }
    STAMP(28);
    const __amdgpu_buffer_rsrc_t rsO = __builtin_amdgcn_make_buffer_rsrc((void*)p.out, 0, MS * NN * 4, 0x00020000);
    const unsigned ooff = ((rb * 16u + (lane >> 4) * 4u) * NN + nj * 64u + (lane & 15) * 4u) * 4u;
    const bool bad = lds_ld_u(lds0 + S_BAD) != 0u;
#pragma unroll
    for (int r = 0; r < 4; ++r) {
        u32x4 o;
#pragma unroll
        for (int i = 0; i < 4; ++i) {
            float y = (float)acc[i][r] * scale;
            float yc = __builtin_amdgcn_fmed3f(y, -60.0f, 60.0f);
            float e = __builtin_amdgcn_exp2f(yc * -1.4426950408889634f);
            o[i] = __builtin_bit_cast(unsigned, y * __builtin_amdgcn_rcpf(1.0f + e));
        }
        if (bad) o = (u32x4){0x7fc00000u, 0x7fc00000u, 0x7fc00000u, 0x7fc00000u};
        __builtin_amdgcn_raw_buffer_store_b128(o, rsO, (int)(ooff + r * (NN * 4)), 0, 17);
    }
    STAMP(29);
}

}

extern "C" void kernel_launch(int rank, int nranks, ncclComm_t comm,
                              hipStream_t stream,
                              void* const* d_in, const int* in_sizes, int n_in,
                              void* d_out, int out_size,
                              const int* in_dtypes, int out_dtype,
                              void* const* d_out_all,
                              void* const* d_ws_all, size_t ws_bytes) {
    (void)comm; (void)in_dtypes; (void)out_dtype; (void)d_out_all;
    if (nranks != NR || (unsigned)rank >= (unsigned)NR || n_in != 4 || ws_bytes < WS_NEEDED ||
        in_sizes[0] != 4096 * KS || in_sizes[1] != KS * NN || out_size != MS * NN) {
        fprintf(stderr, "kernel_launch: unexpected problem shape\n");
        return;
    }
    Params p;
    p.x = (const int8_t*)d_in[0];
    p.w = (const int8_t*)d_in[1];
    p.sx = (const float*)d_in[2];
    p.sw = (const float*)d_in[3];
    p.out = (float*)d_out;
    for (int t = 0; t < NR; ++t) p.ws[t] = (char*)d_ws_all[(rank + t) % NR];
    p.rank = rank;
    p.pad = 0;
    hipLaunchKernelGGL(gemm_ag_kernel, dim3(NWG), dim3(NTHREADS), 0, stream, p);
    HIP_CHECK(hipGetLastError());
}
```

```cpp
#include <hip/hip_runtime.h>
#include <rccl/rccl.h>
#include <cstdio>
#include <cstdint>

#define HIP_CHECK(expr) do { hipError_t _e = (expr); if (_e != hipSuccess) { \
    fprintf(stderr, "HIP error %s at %s:%d\n", hipGetErrorString(_e), __FILE__, __LINE__); \
    return; } } while (0)

namespace {

#define GAS __attribute__((address_space(1)))
#define LAS __attribute__((address_space(3)))
typedef unsigned u32x4 __attribute__((ext_vector_type(4)));
typedef int i32x4 __attribute__((ext_vector_type(4)));
typedef float f32x4 __attribute__((ext_vector_type(4)));

constexpr int NR = 8;
constexpr int MS = 512;
constexpr int KS = 512;
constexpr int NN = 4096;
constexpr int NCH = 8;
constexpr int NWG = 256;
constexpr int NUNIT = NCH * 64;
constexpr int NCW = 10;
constexpr int W_LOAD = 10, W_SEND = 11;
constexpr int NWAVE = 12;
constexpr int NTHREADS = NWAVE * 64;
constexpr int NSEND = 32;
#define WIN 10
#define LD0 12
#define LDN (WIN > 14 ? -1 : 12)
#define XWIN 1
constexpr unsigned NJOB = NUNIT;
static_assert(WIN >= 2 && WIN <= 28 && WIN + 12 <= 63 && LD0 >= 2 && LD0 <= 20, "sender window");
constexpr unsigned TICKETS = NJOB;

constexpr unsigned OFF_CNT = 0;
constexpr unsigned OFF_TMO = 256;
constexpr unsigned OFF_HEAD = 512;
constexpr unsigned OFF_FLAGT = 4096;
constexpr unsigned OFF_FLAGX = 8192;
constexpr unsigned OFF_FLAGW = 16384;
constexpr unsigned OFF_DATA = 1u << 20;
constexpr unsigned W_BYTES = 16u << 20;
constexpr unsigned X_BYTES = 2u << 20;
constexpr unsigned PAR_STRIDE = W_BYTES + X_BYTES;
constexpr size_t WS_NEEDED = (size_t)OFF_DATA + 2 * (size_t)PAR_STRIDE;

constexpr unsigned L_B = 0;
constexpr unsigned L_RING = 0;
constexpr unsigned L_SEQ = 131072;
constexpr unsigned L_SYNC = 131328;
constexpr unsigned S_CALL = L_SYNC, S_RING = L_SYNC + 8, S_TDONE = L_SYNC + 12, S_BAD = L_SYNC + 16;
constexpr unsigned S_DONE = L_SYNC + 32;
constexpr unsigned LDS_BYTES = L_SYNC + 256;

constexpr unsigned long long DEADLINE_TICKS = 100000000ull;

struct Params {
    const int8_t* x;
    const int8_t* w;
    const float* sx;
    const float* sw;
    float* out;
    char* ws[NR];
    int rank;
    int pad;
};

#define STAMP(i) do {} while (0)
#define STAMP_DUMP() do {} while (0)
constexpr unsigned LDS_TOTAL = LDS_BYTES;

template <int N> __device__ __forceinline__ void waitvm() { asm volatile("s_waitcnt vmcnt(%0)" :: "n"(N) : "memory"); }
template <int N> __device__ __forceinline__ void waitvm_n(int n) {
    if (n == 0) waitvm<N>(); else if (n == 1) waitvm<N + 1>(); else if (n == 2) waitvm<N + 2>(); else if (n == 3) waitvm<N + 3>();
    else if (n == 4) waitvm<N + 4>(); else if (n == 5) waitvm<N + 5>(); else if (n == 6) waitvm<N + 6>();
    else if (n == 7) waitvm<N + 7>(); else if (n == 8) waitvm<N + 8>(); else if (n == 9) waitvm<N + 9>();
    else if (n == 10) waitvm<N + 10>(); else if (n == 11) waitvm<N + 11>(); else waitvm<N + 12>();
}
__device__ __forceinline__ void st16_wt(char* base, unsigned off, u32x4 v) {
    asm volatile("global_store_dwordx4 %0, %1, %2 sc0 sc1\n\ts_nop 1" :: "v"(off), "v"(v), "s"(base) : "memory");
}
__device__ __forceinline__ void glds16_sys(const void* gsrc, unsigned lds_dst) {
    unsigned keep;
    asm volatile("s_mov_b32 %0, m0\n\ts_mov_b32 m0, %2\n\ts_nop 0\n\tglobal_load_lds_dwordx4 %1, off sc0 sc1\n\ts_mov_b32 m0, %0"
                 : "=&s"(keep) : "v"(gsrc), "s"(lds_dst) : "memory");
}
__device__ __forceinline__ void glds64_sys(const void* gsrc, unsigned lds_dst) {
    unsigned keep;
    asm volatile("s_mov_b32 %0, m0\n\ts_mov_b32 m0, %2\n\ts_nop 0\n\t"
                 "global_load_lds_dwordx4 %1, off sc0 sc1\n\tglobal_load_lds_dwordx4 %1, off offset:1024 sc0 sc1\n\t"
                 "global_load_lds_dwordx4 %1, off offset:2048 sc0 sc1\n\tglobal_load_lds_dwordx4 %1, off offset:3072 sc0 sc1\n\t"
                 "s_mov_b32 m0, %0"
                 : "=&s"(keep) : "v"(gsrc), "s"(lds_dst) : "memory");
}
__device__ __forceinline__ void lds_rd4(u32x4 (&f)[4], unsigned addr) {
    asm volatile("ds_read_b128 %0, %4\n\tds_read_b128 %1, %4 offset:1024\n\tds_read_b128 %2, %4 offset:2048\n\t"
                 "ds_read_b128 %3, %4 offset:3072\n\ts_waitcnt lgkmcnt(0)"
                 : "=&v"(f[0]), "=&v"(f[1]), "=&v"(f[2]), "=&v"(f[3]) : "v"(addr) : "memory");
}
__device__ __forceinline__ u32x4 lds_rd1(unsigned addr) {
    u32x4 f;
    asm volatile("ds_read_b128 %0, %1\n\ts_waitcnt lgkmcnt(0)" : "=v"(f) : "v"(addr) : "memory");
    return f;
}
__device__ __forceinline__ void ticket_issue(unsigned& q, GAS unsigned* head) {
    unsigned zero = 0u, one = 1u;
    asm volatile("global_atomic_add %0, %1, %2, %3 sc0" : "=v"(q) : "v"(zero), "v"(one), "s"(head) : "memory");
}
__device__ __forceinline__ void lds_add(unsigned addr, unsigned v) {
    asm volatile("ds_add_u32 %0, %1" :: "v"(addr), "v"(v) : "memory");
}
__device__ __forceinline__ void lds_st(unsigned addr, unsigned v) {
    asm volatile("ds_write_b32 %0, %1" :: "v"(addr), "v"(v) : "memory");
}
__device__ __forceinline__ unsigned lds_ld(unsigned addr) {
    unsigned r;
    asm volatile("ds_read_b32 %0, %1\n\ts_waitcnt lgkmcnt(0)" : "=v"(r) : "v"(addr) : "memory");
    return r;
}
__device__ __forceinline__ unsigned lds_ld_u(unsigned addr) { return (unsigned)__builtin_amdgcn_readfirstlane(lds_ld(addr)); }
__device__ __forceinline__ void lds_wait_ge(unsigned addr, unsigned want) {
    while (lds_ld_u(addr) < want) __builtin_amdgcn_s_sleep(1);
}

__device__ __forceinline__ bool give_up(unsigned long long t0, unsigned spins, GAS unsigned* tmo, unsigned call) {
    bool g = __builtin_amdgcn_s_memrealtime() - t0 > DEADLINE_TICKS;
    if ((spins & 15u) == 15u)
        g |= (unsigned)__builtin_amdgcn_readfirstlane(
                 __hip_atomic_load(tmo, __ATOMIC_RELAXED, __HIP_MEMORY_SCOPE_AGENT)) == call;
    if (g) __hip_atomic_store(tmo, call, __ATOMIC_RELAXED, __HIP_MEMORY_SCOPE_AGENT);
    return g;
}
__device__ __forceinline__ void wait_flags(GAS unsigned* flag, unsigned want, GAS unsigned* tmo, unsigned call) {
    unsigned long long t0 = __builtin_amdgcn_s_memrealtime();
    for (unsigned spins = 0;; ++spins) {
        unsigned v = __hip_atomic_load(flag, __ATOMIC_RELAXED, __HIP_MEMORY_SCOPE_SYSTEM);
        if (__all((int)(v - want) >= 0)) break;
        if (give_up(t0, spins, tmo, call)) break;
        __builtin_amdgcn_s_sleep(2);
    }
}

__device__ __forceinline__ unsigned unit_order(unsigned u, unsigned rank) {
    return (u & ~63u) | ((u + 8u * rank) & 63u);
}

template <int LDP, bool FIRST>
__device__ __forceinline__ void send_unit(char* const (&dst)[NR - 1], unsigned woff, const u32x4 (&f)[4], const char* nsrc,
                                          unsigned nslot, unsigned& qn, GAS unsigned* head, GAS unsigned* fprev, bool fact,
                                          unsigned call, unsigned tdone_addr, int lane) {
    if (LDP < 0) {
        if (lane == 0) ticket_issue(qn, head);
        glds64_sys(nsrc, nslot);
    }
#pragma unroll
    for (int k = 0; k < 28; ++k) {
        st16_wt(dst[k % 7], woff + (k / 7) * 1024u, f[k / 7]);
        constexpr int pos[6] = {WIN - 1, LDP, LDP, LDP, LDP, LDP};
        int n = 0;
#pragma unroll
        for (int e = 0; e < 6; ++e)
            n += ((pos[e] >= k - WIN && pos[e] <= k - 1) ? 1 : 0) + ((pos[e] - 28 >= k - WIN) ? 1 : 0);
        waitvm_n<WIN>(n);
        if (k == WIN - 1) {
            waitvm<WIN>();
            if (fact) __hip_atomic_store(fprev, call, __ATOMIC_RELAXED, __HIP_MEMORY_SCOPE_SYSTEM);
        }
        if (k == LDP) {
            if (FIRST) lds_wait_ge(tdone_addr, 1u);
            if (lane == 0) ticket_issue(qn, head);
            glds64_sys(nsrc, nslot);
        }
    }
    constexpr int younger = (27 - LDP) + (WIN - 1 > LDP ? 1 : 0);
    waitvm<(younger < 63 ? younger : 63)>();
    asm volatile("" : "+v"(qn));
}

__global__ void __launch_bounds__(NTHREADS, 1) gemm_ag_kernel(Params p) {
    __shared__ __attribute__((aligned(16))) char smem[LDS_TOTAL];
    const int tid = threadIdx.x;
    const int lane = tid & 63;
    const int wv = __builtin_amdgcn_readfirstlane(tid >> 6);
    const unsigned g = blockIdx.x;
    const unsigned lds0 = (unsigned)(size_t)(LAS char*)smem;
    const bool send_wg = g < NSEND;
    char* const myws = p.ws[0];
    GAS unsigned* const tmo = (GAS unsigned*)(myws + OFF_TMO);
    STAMP(0);

    const unsigned xrb = g >> 3;
    const unsigned xrel = (g + (g >> 3)) & 7u;
    u32x4 xv[NCH];
    unsigned tr[16];
    const unsigned tu = unit_order(g + 256u * wv, p.rank);
    unsigned q0 = 0;
    GAS unsigned* const head = (GAS unsigned*)(myws + OFF_HEAD);

    if (wv == W_SEND) {
        const __amdgpu_buffer_rsrc_t rs = __builtin_amdgcn_make_buffer_rsrc((void*)p.x, 0, 4096 * KS, 0x00020000);
        const unsigned d = (p.rank + xrel) & 7u;
#pragma unroll
        for (int j = 0; j < NCH; ++j)
            xv[j] = __builtin_amdgcn_raw_buffer_load_b128(
                rs, (int)((d * MS + xrb * 16u + (lane & 15)) * KS + j * 64u + (lane >> 4) * 16u), 0, 0);
        if (send_wg && lane == 0) ticket_issue(q0, head);
    } else if (wv == W_LOAD) {
        unsigned c = 0;
        if (lane == 0) c = __hip_atomic_load((GAS unsigned*)(myws + OFF_CNT), __ATOMIC_RELAXED, __HIP_MEMORY_SCOPE_AGENT);
        if (lane < NCH) *(LAS unsigned*)(smem + L_SEQ + lane * 4) = 0u;
        if (lane >= 1 && lane < 16) *(LAS unsigned*)(smem + L_SYNC + lane * 4) = 0u;
        if (lane == 0) *(LAS unsigned*)(smem + S_CALL) = c / NWG + 1u;
        asm volatile("s_waitcnt lgkmcnt(0)" ::: "memory");
    } else if (wv < 2) {
        const unsigned u = tu;
        const __amdgpu_buffer_rsrc_t rs = __builtin_amdgcn_make_buffer_rsrc((void*)p.w, 0, KS * NN, 0x00020000);
#pragma unroll
        for (int t = 0; t < 16; ++t)
            tr[t] = __builtin_amdgcn_raw_buffer_load_b32(
                rs, (int)(((u >> 6) * 64u + (lane >> 4) * 16u + t) * NN + (u & 63u) * 64u + (lane & 15) * 4u), 0, 0);
    }
    __builtin_amdgcn_s_barrier();
    asm volatile("" ::: "memory");
    const unsigned call = lds_ld_u(lds0 + S_CALL);
    const unsigned par = OFF_DATA + (call & 1u) * PAR_STRIDE;
    const unsigned ci = (g - NSEND) >> 3;
    const unsigned cp = (g & 7u) * 4u + ci / 7u;
    const unsigned rg = ci % 7u;
    const unsigned nrb = rg < 4u ? 5u : 4u;
    const unsigned rb0 = rg < 4u ? rg * 5u : 20u + (rg - 4u) * 4u;
    STAMP(1);

    if (wv == W_SEND) {
        __builtin_amdgcn_s_setprio(2);
        char* const xd = p.ws[xrel];
        const unsigned xoff = par + W_BYTES + (p.rank * 32u + xrb) * 8192u + lane * 16u;
        waitvm<0>();
#pragma unroll
        for (int j = 0; j < NCH; ++j) asm volatile("" : "+v"(xv[j]));
        asm volatile("" : "+v"(q0));
        STAMP(2);
#pragma unroll
        for (int j = 0; j < NCH; ++j) {
            st16_wt(xd, xoff + j * 1024u, xv[j]);
            if (j < NCH - 1) waitvm<XWIN>();
        }
        GAS unsigned* const fx = (GAS unsigned*)(xd + OFF_FLAGX + xrb * 128u + p.rank * 4u);
        if (!send_wg) {
            waitvm<0>();
            if (lane == 0) __hip_atomic_store(fx, call, __ATOMIC_RELAXED, __HIP_MEMORY_SCOPE_SYSTEM);
            STAMP(3);
            STAMP_DUMP();
            return;
        }
        STAMP(3);
        char* peer = p.ws[0];
#pragma unroll
        for (int t = 1; t < NR; ++t) peer = ((lane & 7) == t) ? p.ws[t] : peer;
        char* dst[NR - 1];
#pragma unroll
        for (int t = 0; t < NR - 1; ++t) dst[t] = p.ws[1 + (t + g) % 7u];
        const unsigned qbase = (call - 1u) * TICKETS - (unsigned)NSEND;
        unsigned jcur = g;
        unsigned jnxt = (unsigned)__builtin_amdgcn_readfirstlane(q0) - qbase;
        const unsigned ring = (unsigned)__builtin_amdgcn_readfirstlane(lds0 + L_RING);
        const unsigned wbase = par + p.rank * (NUNIT * 4096u);
        const char* wloc = myws + wbase + lane * 16;
        GAS unsigned* fprev = fx;
        bool fact = lane == 0;
        lds_wait_ge(lds0 + S_RING, 1u);
        unsigned slot = 0;
        bool first = true;
#define JOB_UNIT(jb) unit_order((jb), p.rank)
#define JOB_FLAG(jb) ((GAS unsigned*)(peer + OFF_FLAGW + (JOB_UNIT(jb) * 8u + p.rank) * 4u))
        while (jnxt < NJOB) {
            const char* nsrc = wloc + JOB_UNIT(jnxt) * 4096u;
            const unsigned nslot = ring + (slot ^ 1u) * 4096u;
            unsigned qn;
            u32x4 f[4];
            lds_rd4(f, ring + slot * 4096u + lane * 16u);
            const unsigned woff = wbase + JOB_UNIT(jcur) * 4096u + lane * 16u;
            STAMP(4 + (jcur >> 6));
            if (first) send_unit<LD0, true>(dst, woff, f, nsrc, nslot, qn, head, fprev, fact, call, lds0 + S_TDONE, lane);
            else send_unit<LDN, false>(dst, woff, f, nsrc, nslot, qn, head, fprev, fact, call, 0u, lane);
            fprev = JOB_FLAG(jcur);
            fact = lane >= 1 && lane < NR;
            first = false;
            jcur = jnxt;
            jnxt = (unsigned)__builtin_amdgcn_readfirstlane(qn) - qbase;
            slot ^= 1u;
        }
        {
            u32x4 f[4];
            lds_rd4(f, ring + slot * 4096u + lane * 16u);
            const unsigned woff = wbase + JOB_UNIT(jcur) * 4096u + lane * 16u;
#pragma unroll
            for (int k = 0; k < 28; ++k) {
                st16_wt(dst[k % 7], woff + (k / 7) * 1024u, f[k / 7]);
                if (k == WIN - 1) {
                    waitvm<WIN>();
                    if (fact) __hip_atomic_store(fprev, call, __ATOMIC_RELAXED, __HIP_MEMORY_SCOPE_SYSTEM);
                }
            }
        }
        fprev = JOB_FLAG(jcur);
        STAMP(12);
        waitvm<0>();
        STAMP(13);
        if (lane >= 1 && lane < NR) __hip_atomic_store(fprev, call, __ATOMIC_RELAXED, __HIP_MEMORY_SCOPE_SYSTEM);
        STAMP_DUMP();
        return;
    }

    if (wv == W_LOAD) {
        __builtin_amdgcn_s_setprio(3);
        if (send_wg) {
            const char* ft = myws + OFF_FLAGT + lane * 16;
            unsigned long long t0 = __builtin_amdgcn_s_memrealtime();
            for (unsigned spins = 0;; ++spins) {
                u32x4 a, b;
                asm volatile("global_load_dwordx4 %0, %2, off sc0 sc1\n\tglobal_load_dwordx4 %1, %2, off offset:1024 sc0 sc1\n\t"
                             "s_waitcnt vmcnt(0)" : "=&v"(a), "=&v"(b) : "v"(ft) : "memory");
                bool ok = true;
#pragma unroll
                for (int e = 0; e < 4; ++e) ok = ok && (int)(a[e] - call) >= 0 && (int)(b[e] - call) >= 0;
                if (__all(ok)) break;
                if (give_up(t0, spins, tmo, call)) break;
                __builtin_amdgcn_s_sleep(2);
            }
            if (lane == 0) lds_st(lds0 + S_TDONE, 1u);
            STAMP(12);
            if (lane == 0)
                (void)__hip_atomic_fetch_add((GAS unsigned*)(myws + OFF_CNT), 1u, __ATOMIC_RELAXED, __HIP_MEMORY_SCOPE_AGENT);
            STAMP_DUMP();
            return;
        }
        const char* wsrc = myws + par + cp * 8192u + lane * 16;
        const unsigned lbase = (unsigned)__builtin_amdgcn_readfirstlane(lds0 + L_B);
        const unsigned nmfma = 2u * nrb;
        unsigned cs0 = 0, cs1 = 1;
        bool act0 = false, act1 = false;
        unsigned pub0 = 0, pub1 = 0, tmo_now = 0;
        unsigned pending = 0, issued = 0;
        bool counted = false;
        const unsigned long long t0 = __builtin_amdgcn_s_memrealtime();
        for (unsigned spins = 0;; ++spins) {
            if (!act0 && cs0 < NCH && (cs0 < 2 || lds_ld_u(lds0 + S_DONE + (cs0 - 2) * 4u) >= nmfma)) {
                act0 = true; pub0 = 0; pending |= 0xffffu; STAMP(16 + cs0);
            }
            if (!act1 && cs1 < NCH && (cs1 < 2 || lds_ld_u(lds0 + S_DONE + (cs1 - 2) * 4u) >= nmfma)) {
                act1 = true; pub1 = 0; pending |= 0xffff0000u; STAMP(16 + cs1);
            }
            unsigned ready = 0;
            if (pending) {
                const unsigned cl = ((lane & 16) ? cs1 : cs0) & 7u;
                const unsigned fo = OFF_FLAGW + ((cl * 64u + cp * 2u + (lane & 1)) * 8u + ((lane >> 1) & 7)) * 4u;
                unsigned v = __hip_atomic_load((GAS unsigned*)(myws + fo), __ATOMIC_RELAXED, __HIP_MEMORY_SCOPE_SYSTEM);
                ready = (unsigned)__ballot((int)(v - call) >= 0) & pending;
                if (!ready && give_up(t0, spins, tmo, call)) ready = pending;
            }
            waitvm<0>();
            unsigned both = issued & (issued >> 1) & 0x55555555u;
            both = (both | (both >> 1)) & 0x33333333u;
            both = (both | (both >> 2)) & 0x0f0f0f0fu;
            both = (both | (both >> 4)) & 0x00ff00ffu;
            const unsigned n0 = both & 0xffu, n1 = both >> 16;
            if (!pending && cs0 >= NCH - 2 && cs1 >= NCH - 1 && (act0 || cs0 >= NCH) && (act1 || cs1 >= NCH) && lane == 0)
                lds_st(lds0 + S_BAD, tmo_now == call ? 1u : 0u);
            if (act0 && n0 != pub0) {
                pub0 = n0;
                if (lane == 0) lds_st(lds0 + L_SEQ + cs0 * 4u, pub0);
                if (pub0 == 0xffu) { STAMP(4 + cs0); act0 = false; cs0 += 2; issued &= 0xffff0000u; }
            }
            if (act1 && n1 != pub1) {
                pub1 = n1;
                if (lane == 0) lds_st(lds0 + L_SEQ + cs1 * 4u, pub1);
                if (pub1 == 0xffu) { STAMP(4 + cs1); act1 = false; cs1 += 2; issued &= 0xffffu; }
            }
            if (cs0 >= NCH && cs1 >= NCH) break;
            if (!counted && cs0 >= 2) {
                counted = true;
                if (lane == 0)
                    (void)__hip_atomic_fetch_add((GAS unsigned*)(myws + OFF_CNT), 1u, __ATOMIC_RELAXED, __HIP_MEMORY_SCOPE_AGENT);
            }
            if (!ready) { __builtin_amdgcn_s_sleep(1); continue; }
            pending &= ~ready;
            issued |= ready;
            while (ready) {
                const unsigned b = __builtin_ctz(ready);
                ready &= ready - 1;
                const unsigned c = b >= 16 ? cs1 : cs0, q = (b >> 1) & 7u, h = b & 1u;
                glds64_sys(wsrc + (q * NCH + c) * (64u * 4096u) + h * 4096u, lbase + (c & 1u) * 65536u + q * 8192u + h * 4096u);
            }
            if (!pending && cs0 >= NCH - 2 && cs1 >= NCH - 1)
                tmo_now = (unsigned)__builtin_amdgcn_readfirstlane(__hip_atomic_load(tmo, __ATOMIC_RELAXED, __HIP_MEMORY_SCOPE_AGENT));
        }
        STAMP_DUMP();
        return;
    }

    if (wv < 2) {
        const unsigned u = tu;
        u32x4 f[4];
#pragma unroll
        for (int q = 0; q < 4; ++q) {
            unsigned t0 = __builtin_amdgcn_perm(tr[4 * q + 1], tr[4 * q + 0], 0x05010400u);
            unsigned t1 = __builtin_amdgcn_perm(tr[4 * q + 1], tr[4 * q + 0], 0x07030602u);
            unsigned t2 = __builtin_amdgcn_perm(tr[4 * q + 3], tr[4 * q + 2], 0x05010400u);
            unsigned t3 = __builtin_amdgcn_perm(tr[4 * q + 3], tr[4 * q + 2], 0x07030602u);
            f[0][q] = __builtin_amdgcn_perm(t2, t0, 0x05040100u);
            f[1][q] = __builtin_amdgcn_perm(t2, t0, 0x07060302u);
            f[2][q] = __builtin_amdgcn_perm(t3, t1, 0x05040100u);
            f[3][q] = __builtin_amdgcn_perm(t3, t1, 0x07060302u);
        }
        const unsigned woff = par + (p.rank * NUNIT + u) * 4096u + lane * 16u;
#pragma unroll
        for (int i = 0; i < 4; ++i) st16_wt(myws, woff + i * 1024u, f[i]);
        if (send_wg && wv == 0) {
#pragma unroll
            for (int i = 0; i < 4; ++i) *(LAS u32x4*)(smem + L_RING + i * 1024 + lane * 16) = f[i];
            asm volatile("s_waitcnt lgkmcnt(0)" ::: "memory");
            if (lane == 0) lds_st(lds0 + S_RING, 1u);
        }
        waitvm<0>();
        if (lane == 0) {
            __hip_atomic_store((GAS unsigned*)(myws + OFF_FLAGT + u * 4u), call, __ATOMIC_RELAXED, __HIP_MEMORY_SCOPE_SYSTEM);
            __hip_atomic_store((GAS unsigned*)(myws + OFF_FLAGW + (u * 8u + p.rank) * 4u), call, __ATOMIC_RELAXED, __HIP_MEMORY_SCOPE_SYSTEM);
        }
    }
    if (send_wg || (unsigned)wv >= 2u * nrb) { STAMP_DUMP(); return; }
    const float scale = p.sx[0] * p.sw[0];
    const unsigned half = (unsigned)wv >= nrb ? 1u : 0u;
    const unsigned rb = rb0 + (unsigned)wv - half * nrb;
    const unsigned nj = cp * 2u + half;
    const __amdgpu_buffer_rsrc_t rsX =
        __builtin_amdgcn_make_buffer_rsrc((void*)(myws + par + W_BYTES), 0, (int)X_BYTES, 0x00020000);

    wait_flags((GAS unsigned*)(myws + OFF_FLAGX + rb * 128u + (lane & 7) * 4u), call, tmo, call);

    STAMP(2);
    i32x4 acc[4] = {};
#pragma unroll 1
    for (int j = 0; j < NCH; ++j) {
        u32x4 a[NR];
#pragma unroll
        for (int q = 0; q < NR; ++q)
            a[q] = __builtin_amdgcn_raw_buffer_load_b128(rsX, (int)(((q * 32u + rb) * 8u + j) * 1024u + lane * 16u), 0, 17);
        const char* lb = smem + L_B + (j & 1) * 65536 + half * 4096 + lane * 16;
        unsigned donem = 0;
        while (donem != 0xffu) {
            unsigned m = lds_ld_u(lds0 + L_SEQ + j * 4u) & ~donem;
            if (!m) { __builtin_amdgcn_s_sleep(1); continue; }
            if (donem == 0) STAMP(4 + j * 3);
            donem |= m;
            do {
                const unsigned q = __builtin_ctz(m);
                m &= m - 1;
                i32x4 b[4];
#pragma unroll
                for (int i = 0; i < 4; ++i) b[i] = *(const LAS i32x4*)(lb + q * 8192u + i * 1024);
                u32x4 av;
                switch (q) {
                    case 0: av = a[0]; break; case 1: av = a[1]; break; case 2: av = a[2]; break; case 3: av = a[3]; break;
                    case 4: av = a[4]; break; case 5: av = a[5]; break; case 6: av = a[6]; break; default: av = a[7]; break;
                }
#pragma unroll
                for (int i = 0; i < 4; ++i)
                    acc[i] = __builtin_amdgcn_mfma_i32_16x16x64_i8(__builtin_bit_cast(i32x4, av), b[i], acc[i], 0, 0, 0);
            } while (m);
        }
        STAMP(5 + j * 3);
        if (lane == 0) lds_add(lds0 + S_DONE + j * 4u, 1u);
    }
    STAMP(28);
    const __amdgpu_buffer_rsrc_t rsO = __builtin_amdgcn_make_buffer_rsrc((void*)p.out, 0, MS * NN * 4, 0x00020000);
    const unsigned ooff = ((rb * 16u + (lane >> 4) * 4u) * NN + nj * 64u + (lane & 15) * 4u) * 4u;
    const bool bad = lds_ld_u(lds0 + S_BAD) != 0u;
#pragma unroll
    for (int r = 0; r < 4; ++r) {
        u32x4 o;
#pragma unroll
        for (int i = 0; i < 4; ++i) {
            float y = (float)acc[i][r] * scale;
            float yc = __builtin_amdgcn_fmed3f(y, -60.0f, 60.0f);
            float e = __builtin_amdgcn_exp2f(yc * -1.4426950408889634f);
            o[i] = __builtin_bit_cast(unsigned, y * __builtin_amdgcn_rcpf(1.0f + e));
        }
        if (bad) o = (u32x4){0x7fc00000u, 0x7fc00000u, 0x7fc00000u, 0x7fc00000u};
        __builtin_amdgcn_raw_buffer_store_b128(o, rsO, (int)(ooff + r * (NN * 4)), 0, 17);
    }
    STAMP(29);
}

}

extern "C" void kernel_launch(int rank, int nranks, ncclComm_t comm,
                              hipStream_t stream,
                              void* const* d_in, const int* in_sizes, int n_in,
                              void* d_out, int out_size,
                              const int* in_dtypes, int out_dtype,
                              void* const* d_out_all,
                              void* const* d_ws_all, size_t ws_bytes) {
    (void)comm; (void)in_dtypes; (void)out_dtype; (void)d_out_all;
    if (nranks != NR || (unsigned)rank >= (unsigned)NR || n_in != 4 || ws_bytes < WS_NEEDED ||
        in_sizes[0] != 4096 * KS || in_sizes[1] != KS * NN || out_size != MS * NN) {
        fprintf(stderr, "kernel_launch: unexpected problem shape\n");
        return;
    }
    Params p;
    p.x = (const int8_t*)d_in[0];
    p.w = (const int8_t*)d_in[1];
    p.sx = (const float*)d_in[2];
    p.sw = (const float*)d_in[3];
    p.out = (float*)d_out;
    for (int t = 0; t < NR; ++t) p.ws[t] = (char*)d_ws_all[(rank + t) % NR];
    p.rank = rank;
    p.pad = 0;
    hipLaunchKernelGGL(gemm_ag_kernel, dim3(NWG), dim3(NTHREADS), 0, stream, p);
    HIP_CHECK(hipGetLastError());
}
```

```cpp
#include <hip/hip_runtime.h>
#include <rccl/rccl.h>
#include <cstdio>
#include <cstdint>

#define HIP_CHECK(expr) do { hipError_t _e = (expr); if (_e != hipSuccess) { \
    fprintf(stderr, "HIP error %s at %s:%d\n", hipGetErrorString(_e), __FILE__, __LINE__); \
    return; } } while (0)

namespace {

#define GAS __attribute__((address_space(1)))
#define LAS __attribute__((address_space(3)))
typedef unsigned u32x4 __attribute__((ext_vector_type(4)));
typedef int i32x4 __attribute__((ext_vector_type(4)));
typedef float f32x4 __attribute__((ext_vector_type(4)));
typedef float f32x2 __attribute__((ext_vector_type(2)));
typedef unsigned u32x32 __attribute__((ext_vector_type(32)));

constexpr int NR = 8;
constexpr int MS = 512;
constexpr int KS = 512;
constexpr int NN = 4096;
constexpr int NCH = 8;
constexpr int NWG = 256;
constexpr int NUNIT = NCH * 64;
constexpr int NCW = 10;
constexpr int W_LOAD = 10, W_SEND = 11;
constexpr int NWAVE = 12;
constexpr int NTHREADS = NWAVE * 64;
constexpr int NSEND = 32;
#define WIN 10
#define LD0 12
#define LDN (WIN > 14 ? -1 : 12)
#define XWIN 1
constexpr unsigned NJOB = NUNIT;
static_assert(WIN >= 2 && WIN <= 28 && WIN + 12 <= 63 && LD0 >= 2 && LD0 <= 20, "sender window");
constexpr unsigned TICKETS = NJOB;

constexpr unsigned OFF_CNT = 0;
constexpr unsigned OFF_TMO = 256;
constexpr unsigned OFF_HEAD = 512;
constexpr unsigned OFF_FLAGT = 4096;
constexpr unsigned OFF_FLAGX = 8192;
constexpr unsigned OFF_FLAGW = 16384;
constexpr unsigned OFF_DATA = 1u << 20;
constexpr unsigned W_BYTES = 16u << 20;
constexpr unsigned X_BYTES = 2u << 20;
constexpr unsigned PAR_STRIDE = W_BYTES + X_BYTES;
constexpr size_t WS_NEEDED = (size_t)OFF_DATA + 2 * (size_t)PAR_STRIDE;

constexpr unsigned L_B = 0;
constexpr unsigned L_RING = 0;
constexpr unsigned L_SEQ = 131072;
constexpr unsigned L_SYNC = 131328;
constexpr unsigned S_CALL = L_SYNC, S_RING = L_SYNC + 8, S_TDONE = L_SYNC + 12;
constexpr unsigned S_DONE = L_SYNC + 32;
constexpr unsigned LDS_BYTES = L_SYNC + 256;

constexpr unsigned long long DEADLINE_TICKS = 100000000ull;

struct Params {
    const int8_t* x;
    const int8_t* w;
    const float* sx;
    const float* sw;
    float* out;
    char* ws[NR];
    int rank;
    int pad;
};

#define STAMP(i) do {} while (0)
#define STAMP_DUMP() do {} while (0)
constexpr unsigned LDS_TOTAL = LDS_BYTES;

template <int N> __device__ __forceinline__ void waitvm() { asm volatile("s_waitcnt vmcnt(%0)" :: "n"(N) : "memory"); }
template <int N> __device__ __forceinline__ void waitvm_n(int n) {
    if (n == 0) waitvm<N>(); else if (n == 1) waitvm<N + 1>(); else if (n == 2) waitvm<N + 2>(); else if (n == 3) waitvm<N + 3>();
    else if (n == 4) waitvm<N + 4>(); else if (n == 5) waitvm<N + 5>(); else if (n == 6) waitvm<N + 6>();
    else if (n == 7) waitvm<N + 7>(); else if (n == 8) waitvm<N + 8>(); else if (n == 9) waitvm<N + 9>();
    else if (n == 10) waitvm<N + 10>(); else if (n == 11) waitvm<N + 11>(); else waitvm<N + 12>();
}
__device__ __forceinline__ void st16_wt(char* base, unsigned off, u32x4 v) {
    asm volatile("global_store_dwordx4 %0, %1, %2 sc0 sc1\n\ts_nop 1" :: "v"(off), "v"(v), "s"(base) : "memory");
}
__device__ __forceinline__ void glds16_sys(const void* gsrc, unsigned lds_dst) {
    unsigned keep;
    asm volatile("s_mov_b32 %0, m0\n\ts_mov_b32 m0, %2\n\ts_nop 0\n\tglobal_load_lds_dwordx4 %1, off sc0 sc1\n\ts_mov_b32 m0, %0"
                 : "=&s"(keep) : "v"(gsrc), "s"(lds_dst) : "memory");
}
__device__ __forceinline__ void glds64_sys(const void* gsrc, unsigned lds_dst) {
    unsigned keep;
    asm volatile("s_mov_b32 %0, m0\n\ts_mov_b32 m0, %2\n\ts_nop 0\n\t"
                 "global_load_lds_dwordx4 %1, off sc0 sc1\n\tglobal_load_lds_dwordx4 %1, off offset:1024 sc0 sc1\n\t"
                 "global_load_lds_dwordx4 %1, off offset:2048 sc0 sc1\n\tglobal_load_lds_dwordx4 %1, off offset:3072 sc0 sc1\n\t"
                 "s_mov_b32 m0, %0"
                 : "=&s"(keep) : "v"(gsrc), "s"(lds_dst) : "memory");
}
__device__ __forceinline__ void lds_rd4(u32x4 (&f)[4], unsigned addr) {
    asm volatile("ds_read_b128 %0, %4\n\tds_read_b128 %1, %4 offset:1024\n\tds_read_b128 %2, %4 offset:2048\n\t"
                 "ds_read_b128 %3, %4 offset:3072\n\ts_waitcnt lgkmcnt(0)"
                 : "=&v"(f[0]), "=&v"(f[1]), "=&v"(f[2]), "=&v"(f[3]) : "v"(addr) : "memory");
}
__device__ __forceinline__ u32x4 lds_rd1(unsigned addr) {
    u32x4 f;
    asm volatile("ds_read_b128 %0, %1\n\ts_waitcnt lgkmcnt(0)" : "=v"(f) : "v"(addr) : "memory");
    return f;
}
__device__ __forceinline__ void ticket_issue(unsigned& q, GAS unsigned* head) {
    unsigned zero = 0u, one = 1u;
    asm volatile("global_atomic_add %0, %1, %2, %3 sc0" : "=v"(q) : "v"(zero), "v"(one), "s"(head) : "memory");
}
__device__ __forceinline__ void lds_add(unsigned addr, unsigned v) {
    asm volatile("ds_add_u32 %0, %1" :: "v"(addr), "v"(v) : "memory");
}
__device__ __forceinline__ void lds_st(unsigned addr, unsigned v) {
    asm volatile("ds_write_b32 %0, %1" :: "v"(addr), "v"(v) : "memory");
}
__device__ __forceinline__ unsigned lds_ld(unsigned addr) {
    unsigned r;
    asm volatile("ds_read_b32 %0, %1\n\ts_waitcnt lgkmcnt(0)" : "=v"(r) : "v"(addr) : "memory");
    return r;
}
__device__ __forceinline__ unsigned lds_ld_u(unsigned addr) { return (unsigned)__builtin_amdgcn_readfirstlane(lds_ld(addr)); }
__device__ __forceinline__ void lds_wait_ge(unsigned addr, unsigned want) {
    while (lds_ld_u(addr) < want) __builtin_amdgcn_s_sleep(1);
}

__device__ __forceinline__ bool give_up(unsigned long long t0, unsigned spins, GAS unsigned* tmo, unsigned call) {
    bool g = __builtin_amdgcn_s_memrealtime() - t0 > DEADLINE_TICKS;
    if ((spins & 15u) == 15u)
        g |= (unsigned)__builtin_amdgcn_readfirstlane(
                 __hip_atomic_load(tmo, __ATOMIC_RELAXED, __HIP_MEMORY_SCOPE_AGENT)) == call;
    if (g) __hip_atomic_store(tmo, call, __ATOMIC_RELAXED, __HIP_MEMORY_SCOPE_AGENT);
    return g;
}
__device__ __forceinline__ void wait_flags(GAS unsigned* flag, unsigned want, GAS unsigned* tmo, unsigned call) {
    unsigned long long t0 = __builtin_amdgcn_s_memrealtime();
    for (unsigned spins = 0;; ++spins) {
        unsigned v = __hip_atomic_load(flag, __ATOMIC_RELAXED, __HIP_MEMORY_SCOPE_SYSTEM);
        if (__all((int)(v - want) >= 0)) break;
        if (give_up(t0, spins, tmo, call)) break;
        __builtin_amdgcn_s_sleep(2);
    }
}

__device__ __forceinline__ unsigned unit_order(unsigned u, unsigned rank) {
    return (u & ~63u) | ((u + 8u * rank) & 63u);
}

template <int LDP, bool FIRST>
__device__ __forceinline__ void send_unit(char* const (&dst)[NR - 1], unsigned woff, const u32x4 (&f)[4], const char* nsrc,
                                          unsigned nslot, unsigned& qn, GAS unsigned* head, GAS unsigned* fprev, bool fact,
                                          unsigned call, unsigned tdone_addr, int lane) {
    if (LDP < 0) {
        if (lane == 0) ticket_issue(qn, head);
        glds64_sys(nsrc, nslot);
    }
#pragma unroll
    for (int k = 0; k < 28; ++k) {
        st16_wt(dst[k % 7], woff + (k / 7) * 1024u, f[k / 7]);
        constexpr int pos[6] = {WIN - 1, LDP, LDP, LDP, LDP, LDP};
        int n = 0;
#pragma unroll
        for (int e = 0; e < 6; ++e)
            n += ((pos[e] >= k - WIN && pos[e] <= k - 1) ? 1 : 0) + ((pos[e] - 28 >= k - WIN) ? 1 : 0);
        waitvm_n<WIN>(n);
        if (k == WIN - 1) {
            waitvm<WIN>();
            if (fact) __hip_atomic_store(fprev, call, __ATOMIC_RELAXED, __HIP_MEMORY_SCOPE_SYSTEM);
        }
        if (k == LDP) {
            if (FIRST) lds_wait_ge(tdone_addr, 1u);
            if (lane == 0) ticket_issue(qn, head);
            glds64_sys(nsrc, nslot);
        }
    }
    constexpr int younger = (27 - LDP) + (WIN - 1 > LDP ? 1 : 0);
    waitvm<(younger < 63 ? younger : 63)>();
    asm volatile("" : "+v"(qn));
}

__global__ void __launch_bounds__(NTHREADS, 1) gemm_ag_kernel(Params p) {
    __shared__ __attribute__((aligned(16))) char smem[LDS_TOTAL];
    const int tid = threadIdx.x;
    const int lane = tid & 63;
    const int wv = __builtin_amdgcn_readfirstlane(tid >> 6);
    const unsigned g = blockIdx.x;
    const unsigned lds0 = (unsigned)(size_t)(LAS char*)smem;
    const bool send_wg = g < NSEND;
    char* const myws = p.ws[0];
    GAS unsigned* const tmo = (GAS unsigned*)(myws + OFF_TMO);
    STAMP(0);

    const unsigned xrb = g >> 3;
    const unsigned xrel = (g + (g >> 3)) & 7u;
    u32x4 xv[NCH];
    unsigned tr[16];
    const unsigned tu = unit_order(g + 256u * wv, p.rank);
    unsigned q0 = 0;
    GAS unsigned* const head = (GAS unsigned*)(myws + OFF_HEAD);

    if (wv == W_SEND) {
        const __amdgpu_buffer_rsrc_t rs = __builtin_amdgcn_make_buffer_rsrc((void*)p.x, 0, 4096 * KS, 0x00020000);
        const unsigned d = (p.rank + xrel) & 7u;
#pragma unroll
        for (int j = 0; j < NCH; ++j)
            xv[j] = __builtin_amdgcn_raw_buffer_load_b128(
                rs, (int)((d * MS + xrb * 16u + (lane & 15)) * KS + j * 64u + (lane >> 4) * 16u), 0, 0);
        if (send_wg && lane == 0) ticket_issue(q0, head);
    } else if (wv == W_LOAD) {
        unsigned c = 0;
        if (lane == 0) c = __hip_atomic_load((GAS unsigned*)(myws + OFF_CNT), __ATOMIC_RELAXED, __HIP_MEMORY_SCOPE_AGENT);
        if (lane < NCH) *(LAS unsigned*)(smem + L_SEQ + lane * 4) = 0u;
        if (lane >= 1 && lane < 16) *(LAS unsigned*)(smem + L_SYNC + lane * 4) = 0u;
        if (lane == 0) *(LAS unsigned*)(smem + S_CALL) = c / NWG + 1u;
        asm volatile("s_waitcnt lgkmcnt(0)" ::: "memory");
    } else if (wv < 2) {
        const unsigned u = tu;
        const __amdgpu_buffer_rsrc_t rs = __builtin_amdgcn_make_buffer_rsrc((void*)p.w, 0, KS * NN, 0x00020000);
#pragma unroll
        for (int t = 0; t < 16; ++t)
            tr[t] = __builtin_amdgcn_raw_buffer_load_b32(
                rs, (int)(((u >> 6) * 64u + (lane >> 4) * 16u + t) * NN + (u & 63u) * 64u + (lane & 15) * 4u), 0, 0);
    }
    __builtin_amdgcn_s_barrier();
    asm volatile("" ::: "memory");
    const unsigned call = lds_ld_u(lds0 + S_CALL);
    const unsigned par = OFF_DATA + (call & 1u) * PAR_STRIDE;
    const unsigned ci = (g - NSEND) >> 3;
    const unsigned cp = (g & 7u) * 4u + ci / 7u;
    const unsigned rg = ci % 7u;
    const unsigned nrb = rg < 4u ? 5u : 4u;
    const unsigned rb0 = rg < 4u ? rg * 5u : 20u + (rg - 4u) * 4u;
    STAMP(1);

    if (wv == W_SEND) {
        __builtin_amdgcn_s_setprio(2);
        char* const xd = p.ws[xrel];
        const unsigned xoff = par + W_BYTES + (p.rank * 32u + xrb) * 8192u + lane * 16u;
        waitvm<0>();
#pragma unroll
        for (int j = 0; j < NCH; ++j) asm volatile("" : "+v"(xv[j]));
        asm volatile("" : "+v"(q0));
        STAMP(2);
#pragma unroll
        for (int j = 0; j < NCH; ++j) {
            st16_wt(xd, xoff + j * 1024u, xv[j]);
            if (j < NCH - 1) waitvm<XWIN>();
        }
        GAS unsigned* const fx = (GAS unsigned*)(xd + OFF_FLAGX + xrb * 128u + p.rank * 4u);
        if (!send_wg) {
            waitvm<0>();
            if (lane == 0) __hip_atomic_store(fx, call, __ATOMIC_RELAXED, __HIP_MEMORY_SCOPE_SYSTEM);
            STAMP(3);
            STAMP_DUMP();
            return;
        }
        STAMP(3);
        char* peer = p.ws[0];
#pragma unroll
        for (int t = 1; t < NR; ++t) peer = ((lane & 7) == t) ? p.ws[t] : peer;
        char* dst[NR - 1];
#pragma unroll
        for (int t = 0; t < NR - 1; ++t) dst[t] = p.ws[1 + (t + g) % 7u];
        const unsigned qbase = (call - 1u) * TICKETS - (unsigned)NSEND;
        unsigned jcur = g;
        unsigned jnxt = (unsigned)__builtin_amdgcn_readfirstlane(q0) - qbase;
        const unsigned ring = (unsigned)__builtin_amdgcn_readfirstlane(lds0 + L_RING);
        const unsigned wbase = par + p.rank * (NUNIT * 4096u);
        const char* wloc = myws + wbase + lane * 16;
        GAS unsigned* fprev = fx;
        bool fact = lane == 0;
        lds_wait_ge(lds0 + S_RING, 1u);
        unsigned slot = 0;
        bool first = true;
#define JOB_UNIT(jb) unit_order((jb), p.rank)
#define JOB_FLAG(jb) ((GAS unsigned*)(peer + OFF_FLAGW + (JOB_UNIT(jb) * 8u + p.rank) * 4u))
        while (jnxt < NJOB) {
            const char* nsrc = wloc + JOB_UNIT(jnxt) * 4096u;
            const unsigned nslot = ring + (slot ^ 1u) * 4096u;
            unsigned qn;
            u32x4 f[4];
            lds_rd4(f, ring + slot * 4096u + lane * 16u);
            const unsigned woff = wbase + JOB_UNIT(jcur) * 4096u + lane * 16u;
            STAMP(4 + (jcur >> 6));
            if (first) send_unit<LD0, true>(dst, woff, f, nsrc, nslot, qn, head, fprev, fact, call, lds0 + S_TDONE, lane);
            else send_unit<LDN, false>(dst, woff, f, nsrc, nslot, qn, head, fprev, fact, call, 0u, lane);
            fprev = JOB_FLAG(jcur);
            fact = lane >= 1 && lane < NR;
            first = false;
            jcur = jnxt;
            jnxt = (unsigned)__builtin_amdgcn_readfirstlane(qn) - qbase;
            slot ^= 1u;
        }
        {
            u32x4 f[4];
            lds_rd4(f, ring + slot * 4096u + lane * 16u);
            const unsigned woff = wbase + JOB_UNIT(jcur) * 4096u + lane * 16u;
#pragma unroll
            for (int k = 0; k < 28; ++k) {
                st16_wt(dst[k % 7], woff + (k / 7) * 1024u, f[k / 7]);
                if (k == WIN - 1) {
                    waitvm<WIN>();
                    if (fact) __hip_atomic_store(fprev, call, __ATOMIC_RELAXED, __HIP_MEMORY_SCOPE_SYSTEM);
                }
            }
        }
        fprev = JOB_FLAG(jcur);
        STAMP(12);
        waitvm<0>();
        STAMP(13);
        if (lane >= 1 && lane < NR) __hip_atomic_store(fprev, call, __ATOMIC_RELAXED, __HIP_MEMORY_SCOPE_SYSTEM);
        STAMP_DUMP();
        return;
    }

    if (wv == W_LOAD) {
        __builtin_amdgcn_s_setprio(3);
        if (send_wg) {
            const char* ft = myws + OFF_FLAGT + lane * 16;
            unsigned long long t0 = __builtin_amdgcn_s_memrealtime();
            for (unsigned spins = 0;; ++spins) {
                u32x4 a, b;
                asm volatile("global_load_dwordx4 %0, %2, off sc0 sc1\n\tglobal_load_dwordx4 %1, %2, off offset:1024 sc0 sc1\n\t"
                             "s_waitcnt vmcnt(0)" : "=&v"(a), "=&v"(b) : "v"(ft) : "memory");
                bool ok = true;
#pragma unroll
                for (int e = 0; e < 4; ++e) ok = ok && (int)(a[e] - call) >= 0 && (int)(b[e] - call) >= 0;
                if (__all(ok)) break;
                if (give_up(t0, spins, tmo, call)) break;
                __builtin_amdgcn_s_sleep(2);
            }
            if (lane == 0) lds_st(lds0 + S_TDONE, 1u);
            STAMP(12);
            if (lane == 0)
                (void)__hip_atomic_fetch_add((GAS unsigned*)(myws + OFF_CNT), 1u, __ATOMIC_RELAXED, __HIP_MEMORY_SCOPE_AGENT);
            STAMP_DUMP();
            return;
        }
        const char* wsrc = myws + par + cp * 8192u + lane * 16;
        const unsigned lbase = (unsigned)__builtin_amdgcn_readfirstlane(lds0 + L_B);
        const unsigned nmfma = 2u * nrb;
        unsigned cs0 = 0, cs1 = 1;
        bool act0 = false, act1 = false;
        unsigned pub0 = 0, pub1 = 0, tmo_now = 0;
        unsigned pending = 0, issued = 0;
        bool counted = false;
        const unsigned long long t0 = __builtin_amdgcn_s_memrealtime();
        for (unsigned spins = 0;; ++spins) {
            if (!act0 && cs0 < NCH && (cs0 < 2 || lds_ld_u(lds0 + S_DONE + (cs0 - 2) * 4u) >= nmfma)) {
                act0 = true; pub0 = 0; pending |= 0xffffu; STAMP(16 + cs0);
            }
            if (!act1 && cs1 < NCH && (cs1 < 2 || lds_ld_u(lds0 + S_DONE + (cs1 - 2) * 4u) >= nmfma)) {
                act1 = true; pub1 = 0; pending |= 0xffff0000u; STAMP(16 + cs1);
            }
            unsigned ready = 0;
            if (pending) {
                const unsigned cl = ((lane & 16) ? cs1 : cs0) & 7u;
                const unsigned fo = OFF_FLAGW + ((cl * 64u + cp * 2u + (lane & 1)) * 8u + ((lane >> 1) & 7)) * 4u;
                unsigned v = __hip_atomic_load((GAS unsigned*)(myws + fo), __ATOMIC_RELAXED, __HIP_MEMORY_SCOPE_SYSTEM);
                ready = (unsigned)__ballot((int)(v - call) >= 0) & pending;
                if (!ready && give_up(t0, spins, tmo, call)) ready = pending;
            }
            waitvm<0>();
            unsigned both = issued & (issued >> 1) & 0x55555555u;
            both = (both | (both >> 1)) & 0x33333333u;
            both = (both | (both >> 2)) & 0x0f0f0f0fu;
            both = (both | (both >> 4)) & 0x00ff00ffu;
            const unsigned n0 = both & 0xffu, n1 = both >> 16;
            const unsigned badbit = (!pending && cs0 >= NCH - 2 && cs1 >= NCH - 1 && (act0 || cs0 >= NCH) && (act1 || cs1 >= NCH) &&
                                     tmo_now == call) ? 0x100u : 0u;
            if (act0 && n0 != pub0) {
                pub0 = n0;
                if (lane == 0) lds_st(lds0 + L_SEQ + cs0 * 4u, pub0 | badbit);
                if (pub0 == 0xffu) { STAMP(4 + cs0); act0 = false; cs0 += 2; issued &= 0xffff0000u; }
            }
            if (act1 && n1 != pub1) {
                pub1 = n1;
                if (lane == 0) lds_st(lds0 + L_SEQ + cs1 * 4u, pub1 | badbit);
                if (pub1 == 0xffu) { STAMP(4 + cs1); act1 = false; cs1 += 2; issued &= 0xffffu; }
            }
            if (cs0 >= NCH && cs1 >= NCH) break;
            if (!counted && cs0 >= 2) {
                counted = true;
                if (lane == 0)
                    (void)__hip_atomic_fetch_add((GAS unsigned*)(myws + OFF_CNT), 1u, __ATOMIC_RELAXED, __HIP_MEMORY_SCOPE_AGENT);
            }
            if (!ready) { if (cs1 < NCH - 1) __builtin_amdgcn_s_sleep(1); continue; }
            pending &= ~ready;
            issued |= ready;
            while (ready) {
                const unsigned b = __builtin_ctz(ready);
                ready &= ready - 1;
                const unsigned c = b >= 16 ? cs1 : cs0, q = (b >> 1) & 7u, h = b & 1u;
                glds64_sys(wsrc + (q * NCH + c) * (64u * 4096u) + h * 4096u, lbase + (c & 1u) * 65536u + q * 8192u + h * 4096u);
            }
            if (!pending && cs0 >= NCH - 2 && cs1 >= NCH - 1)
                tmo_now = (unsigned)__builtin_amdgcn_readfirstlane(__hip_atomic_load(tmo, __ATOMIC_RELAXED, __HIP_MEMORY_SCOPE_AGENT));
        }
        STAMP_DUMP();
        return;
    }

    if (wv < 2) {
        const unsigned u = tu;
        u32x4 f[4];
#pragma unroll
        for (int q = 0; q < 4; ++q) {
            unsigned t0 = __builtin_amdgcn_perm(tr[4 * q + 1], tr[4 * q + 0], 0x05010400u);
            unsigned t1 = __builtin_amdgcn_perm(tr[4 * q + 1], tr[4 * q + 0], 0x07030602u);
            unsigned t2 = __builtin_amdgcn_perm(tr[4 * q + 3], tr[4 * q + 2], 0x05010400u);
            unsigned t3 = __builtin_amdgcn_perm(tr[4 * q + 3], tr[4 * q + 2], 0x07030602u);
            f[0][q] = __builtin_amdgcn_perm(t2, t0, 0x05040100u);
            f[1][q] = __builtin_amdgcn_perm(t2, t0, 0x07060302u);
            f[2][q] = __builtin_amdgcn_perm(t3, t1, 0x05040100u);
            f[3][q] = __builtin_amdgcn_perm(t3, t1, 0x07060302u);
        }
        const unsigned woff = par + (p.rank * NUNIT + u) * 4096u + lane * 16u;
#pragma unroll
        for (int i = 0; i < 4; ++i) st16_wt(myws, woff + i * 1024u, f[i]);
        if (send_wg && wv == 0) {
#pragma unroll
            for (int i = 0; i < 4; ++i) *(LAS u32x4*)(smem + L_RING + i * 1024 + lane * 16) = f[i];
            asm volatile("s_waitcnt lgkmcnt(0)" ::: "memory");
            if (lane == 0) lds_st(lds0 + S_RING, 1u);
        }
        waitvm<0>();
        if (lane == 0) {
            __hip_atomic_store((GAS unsigned*)(myws + OFF_FLAGT + u * 4u), call, __ATOMIC_RELAXED, __HIP_MEMORY_SCOPE_SYSTEM);
            __hip_atomic_store((GAS unsigned*)(myws + OFF_FLAGW + (u * 8u + p.rank) * 4u), call, __ATOMIC_RELAXED, __HIP_MEMORY_SCOPE_SYSTEM);
        }
    }
    if (send_wg || (unsigned)wv >= 2u * nrb) { STAMP_DUMP(); return; }
    float scale = p.sx[0] * p.sw[0];
    const unsigned half = (unsigned)wv >= nrb ? 1u : 0u;
    const unsigned rb = rb0 + (unsigned)wv - half * nrb;
    const unsigned nj = cp * 2u + half;
    const __amdgpu_buffer_rsrc_t rsX =
        __builtin_amdgcn_make_buffer_rsrc((void*)(myws + par + W_BYTES), 0, (int)X_BYTES, 0x00020000);

    float c2 = scale * -1.4426950408889634f;
    char* const outp = (char*)p.out;
    unsigned ooff = ((rb * 16u + (lane >> 4) * 4u) * NN + nj * 64u + (lane & 15) * 4u) * 4u;
    asm volatile("" : "+v"(scale), "+v"(c2), "+v"(ooff) : "s"(outp));

    wait_flags((GAS unsigned*)(myws + OFF_FLAGX + rb * 128u + (lane & 7) * 4u), call, tmo, call);

    STAMP(2);
    i32x4 acc[4] = {};
    unsigned bad = 0;
#pragma unroll 1
    for (int j = 0; j < NCH; ++j) {
        u32x32 a;
#pragma unroll
        for (int q = 0; q < NR; ++q) {
            const u32x4 t = __builtin_amdgcn_raw_buffer_load_b128(rsX, (int)(((q * 32u + rb) * 8u + j) * 1024u + lane * 16u), 0, 17);
            a[4 * q] = t[0]; a[4 * q + 1] = t[1]; a[4 * q + 2] = t[2]; a[4 * q + 3] = t[3];
        }
        const char* lb = smem + L_B + (j & 1) * 65536 + half * 4096 + lane * 16;
        unsigned seen = 0, m = 0;
#pragma unroll 1
        for (int left = NR; left > 0;) {
            if (m == 0u) {
                const unsigned word = lds_ld_u(lds0 + L_SEQ + j * 4u);
                bad |= word >> 8;
                m = word & 0xffu & ~seen;
                if (m == 0u) { __builtin_amdgcn_s_sleep(1); continue; }
                if (seen == 0) STAMP(4 + j * 3);
                seen |= m;
            }
            const unsigned q = __builtin_ctz(m);
            m &= m - 1;
            --left;
            i32x4 b[4];
#pragma unroll
            for (int i = 0; i < 4; ++i) b[i] = *(const LAS i32x4*)(lb + q * 8192u + i * 1024);
            const u32x4 av = {a[4 * q], a[4 * q + 1], a[4 * q + 2], a[4 * q + 3]};
#pragma unroll
            for (int i = 0; i < 4; ++i)
                acc[i] = __builtin_amdgcn_mfma_i32_16x16x64_i8(__builtin_bit_cast(i32x4, av), b[i], acc[i], 0, 0, 0);
        }
        STAMP(5 + j * 3);
        if (j < NCH - 2 && lane == 0) lds_add(lds0 + S_DONE + j * 4u, 1u);
    }
    STAMP(28);
    const __amdgpu_buffer_rsrc_t rsO = __builtin_amdgcn_make_buffer_rsrc((void*)outp, 0, MS * NN * 4, 0x00020000);
    if (__builtin_expect(bad != 0u, 0)) {
        const u32x4 nan4 = {0x7fc00000u, 0x7fc00000u, 0x7fc00000u, 0x7fc00000u};
#pragma unroll
        for (int r = 0; r < 4; ++r) __builtin_amdgcn_raw_buffer_store_b128(nan4, rsO, (int)(ooff + r * (NN * 4)), 0, 17);
    } else {
        constexpr float TMAX = 60.0f * 1.4426950408889634f;
#pragma unroll
        for (int r = 0; r < 4; ++r) {
            u32x4 o;
#pragma unroll
            for (int i = 0; i < 4; i += 2) {
                const f32x2 a = {(float)acc[i][r], (float)acc[i + 1][r]};
                const f32x2 y = a * scale;
                f32x2 t = a * c2;
                t[0] = __builtin_amdgcn_exp2f(__builtin_amdgcn_fmed3f(t[0], -TMAX, TMAX));
                t[1] = __builtin_amdgcn_exp2f(__builtin_amdgcn_fmed3f(t[1], -TMAX, TMAX));
                t = t + 1.0f;
                t[0] = __builtin_amdgcn_rcpf(t[0]);
                t[1] = __builtin_amdgcn_rcpf(t[1]);
                const f32x2 v = y * t;
                const float v0 = v[0], v1 = v[1];
                o[i] = __builtin_bit_cast(unsigned, v0);
                o[i + 1] = __builtin_bit_cast(unsigned, v1);
            }
            __builtin_amdgcn_raw_buffer_store_b128(o, rsO, (int)(ooff + r * (NN * 4)), 0, 17);
        }
    }
    STAMP(29);
}

}

extern "C" void kernel_launch(int rank, int nranks, ncclComm_t comm,
                              hipStream_t stream,
                              void* const* d_in, const int* in_sizes, int n_in,
                              void* d_out, int out_size,
                              const int* in_dtypes, int out_dtype,
                              void* const* d_out_all,
                              void* const* d_ws_all, size_t ws_bytes) {
    (void)comm; (void)in_dtypes; (void)out_dtype; (void)d_out_all;
    if (nranks != NR || (unsigned)rank >= (unsigned)NR || n_in != 4 || ws_bytes < WS_NEEDED ||
        in_sizes[0] != 4096 * KS || in_sizes[1] != KS * NN || out_size != MS * NN) {
        fprintf(stderr, "kernel_launch: unexpected problem shape\n");
        return;
    }
    Params p;
    p.x = (const int8_t*)d_in[0];
    p.w = (const int8_t*)d_in[1];
    p.sx = (const float*)d_in[2];
    p.sw = (const float*)d_in[3];
    p.out = (float*)d_out;
    for (int t = 0; t < NR; ++t) p.ws[t] = (char*)d_ws_all[(rank + t) % NR];
    p.rank = rank;
    p.pad = 0;
    hipLaunchKernelGGL(gemm_ag_kernel, dim3(NWG), dim3(NTHREADS), 0, stream, p);
    HIP_CHECK(hipGetLastError());
}
```

```cpp
#include <hip/hip_runtime.h>
#include <rccl/rccl.h>
#include <cstdio>
#include <cstdint>

#define HIP_CHECK(expr) do { hipError_t _e = (expr); if (_e != hipSuccess) { \
    fprintf(stderr, "HIP error %s at %s:%d\n", hipGetErrorString(_e), __FILE__, __LINE__); \
    return; } } while (0)

namespace {

#define GAS __attribute__((address_space(1)))
#define LAS __attribute__((address_space(3)))
typedef unsigned u32x4 __attribute__((ext_vector_type(4)));
typedef int i32x4 __attribute__((ext_vector_type(4)));
typedef float f32x4 __attribute__((ext_vector_type(4)));
typedef float f32x2 __attribute__((ext_vector_type(2)));
typedef unsigned u32x32 __attribute__((ext_vector_type(32)));

constexpr int NR = 8;
constexpr int MS = 512;
constexpr int KS = 512;
constexpr int NN = 4096;
constexpr int NCH = 8;
constexpr int NWG = 256;
constexpr int NUNIT = NCH * 64;
constexpr int NCW = 10;
constexpr int W_LOAD = 10, W_SEND = 11;
constexpr int NWAVE = 12;
constexpr int NTHREADS = NWAVE * 64;
constexpr int NSEND = 32;
#define WIN 10
#define LDN 12
#define XWIN 1
constexpr unsigned NJOB = NUNIT;
static_assert(WIN >= 2 && WIN <= 14 && LDN >= 2 && LDN + WIN <= 27 && 2 * NSEND <= 64, "sender window");
constexpr unsigned TICKETS = NJOB - NSEND;

constexpr unsigned OFF_CNT = 0;
constexpr unsigned OFF_TMO = 256;
constexpr unsigned OFF_HEAD = 512;
constexpr unsigned OFF_FLAGT = 4096;
constexpr unsigned OFF_FLAGX = 8192;
constexpr unsigned OFF_FLAGW = 16384;
constexpr unsigned OFF_DATA = 1u << 20;
constexpr unsigned W_BYTES = 16u << 20;
constexpr unsigned X_BYTES = 2u << 20;
constexpr unsigned PAR_STRIDE = W_BYTES + X_BYTES;
constexpr size_t WS_NEEDED = (size_t)OFF_DATA + 2 * (size_t)PAR_STRIDE;

constexpr unsigned L_B = 0;
constexpr unsigned L_RING = 0;
constexpr unsigned L_SEQ = 131072;
constexpr unsigned L_SYNC = 131328;
constexpr unsigned S_CALL = L_SYNC, S_RING = L_SYNC + 8, S_TDONE = L_SYNC + 12;
constexpr unsigned S_DONE = L_SYNC + 32;
constexpr unsigned LDS_BYTES = L_SYNC + 256;

#define DEADLINE 100000000ull
constexpr unsigned long long DEADLINE_TICKS = DEADLINE;

struct Params {
    const int8_t* x;
    const int8_t* w;
    const float* sx;
    const float* sw;
    float* out;
    char* ws[NR];
    int rank;
    int pad;
};

#define STAMP(i) do {} while (0)
#define STAMP_DUMP() do {} while (0)
constexpr unsigned LDS_TOTAL = LDS_BYTES;

template <int N> __device__ __forceinline__ void waitvm() { asm volatile("s_waitcnt vmcnt(%0)" :: "n"(N) : "memory"); }
__device__ __forceinline__ void st16_wt(char* base, unsigned off, u32x4 v) {
    asm volatile("global_store_dwordx4 %0, %1, %2 sc0 sc1\n\ts_nop 1" :: "v"(off), "v"(v), "s"(base) : "memory");
}
__device__ __forceinline__ void glds16_sys(const void* gsrc, unsigned lds_dst) {
    unsigned keep;
    asm volatile("s_mov_b32 %0, m0\n\ts_mov_b32 m0, %2\n\ts_nop 0\n\tglobal_load_lds_dwordx4 %1, off sc0 sc1\n\ts_mov_b32 m0, %0"
                 : "=&s"(keep) : "v"(gsrc), "s"(lds_dst) : "memory");
}
__device__ __forceinline__ void glds64_sys(const void* gsrc, unsigned lds_dst) {
    unsigned keep;
    asm volatile("s_mov_b32 %0, m0\n\ts_mov_b32 m0, %2\n\ts_nop 0\n\t"
                 "global_load_lds_dwordx4 %1, off sc0 sc1\n\tglobal_load_lds_dwordx4 %1, off offset:1024 sc0 sc1\n\t"
                 "global_load_lds_dwordx4 %1, off offset:2048 sc0 sc1\n\tglobal_load_lds_dwordx4 %1, off offset:3072 sc0 sc1\n\t"
                 "s_mov_b32 m0, %0"
                 : "=&s"(keep) : "v"(gsrc), "s"(lds_dst) : "memory");
}
__device__ __forceinline__ void lds_rd4(u32x4 (&f)[4], unsigned addr) {
    asm volatile("ds_read_b128 %0, %4\n\tds_read_b128 %1, %4 offset:1024\n\tds_read_b128 %2, %4 offset:2048\n\t"
                 "ds_read_b128 %3, %4 offset:3072\n\ts_waitcnt lgkmcnt(0)"
                 : "=&v"(f[0]), "=&v"(f[1]), "=&v"(f[2]), "=&v"(f[3]) : "v"(addr) : "memory");
}
__device__ __forceinline__ u32x4 lds_rd1(unsigned addr) {
    u32x4 f;
    asm volatile("ds_read_b128 %0, %1\n\ts_waitcnt lgkmcnt(0)" : "=v"(f) : "v"(addr) : "memory");
    return f;
}
__device__ __forceinline__ void ticket_issue(unsigned& q, GAS unsigned* head) {
    unsigned zero = 0u, one = 1u;
    asm volatile("global_atomic_add %0, %1, %2, %3 sc0" : "=v"(q) : "v"(zero), "v"(one), "s"(head) : "memory");
}
__device__ __forceinline__ void lds_add(unsigned addr, unsigned v) {
    asm volatile("ds_add_u32 %0, %1" :: "v"(addr), "v"(v) : "memory");
}
__device__ __forceinline__ void lds_st(unsigned addr, unsigned v) {
    asm volatile("ds_write_b32 %0, %1" :: "v"(addr), "v"(v) : "memory");
}
__device__ __forceinline__ unsigned lds_ld(unsigned addr) {
    unsigned r;
    asm volatile("ds_read_b32 %0, %1\n\ts_waitcnt lgkmcnt(0)" : "=v"(r) : "v"(addr) : "memory");
    return r;
}
__device__ __forceinline__ unsigned lds_ld_u(unsigned addr) { return (unsigned)__builtin_amdgcn_readfirstlane(lds_ld(addr)); }
__device__ __forceinline__ void lds_wait_ge(unsigned addr, unsigned want) {
    while (lds_ld_u(addr) < want) __builtin_amdgcn_s_sleep(1);
}

__device__ __forceinline__ bool give_up(unsigned long long t0, unsigned spins, GAS unsigned* tmo, unsigned call) {
    bool g = __builtin_amdgcn_s_memrealtime() - t0 > DEADLINE_TICKS;
    if ((spins & 15u) == 15u)
        g |= (unsigned)__builtin_amdgcn_readfirstlane(
                 __hip_atomic_load(tmo, __ATOMIC_RELAXED, __HIP_MEMORY_SCOPE_AGENT)) == call;
    if (g) __hip_atomic_store(tmo, call, __ATOMIC_RELAXED, __HIP_MEMORY_SCOPE_AGENT);
    return g;
}
__device__ __forceinline__ void wait_flags(GAS unsigned* flag, unsigned want, GAS unsigned* tmo, unsigned call) {
    unsigned long long t0 = __builtin_amdgcn_s_memrealtime();
    for (unsigned spins = 0;; ++spins) {
        unsigned v = __hip_atomic_load(flag, __ATOMIC_RELAXED, __HIP_MEMORY_SCOPE_SYSTEM);
        if (__all((int)(v - want) >= 0)) break;
        if (give_up(t0, spins, tmo, call)) break;
        __builtin_amdgcn_s_sleep(4);
    }
}

__device__ __forceinline__ unsigned unit_order(unsigned u, unsigned rank) {
    return (u & ~63u) | ((u + 8u * rank) & 63u);
}

template <int N> __device__ __forceinline__ void waitvm_unless(unsigned skip) {
    asm volatile("s_cmp_lg_u32 %0, 0\n\ts_cbranch_scc1 .Lnowait%=\n\ts_waitcnt vmcnt(%1)\n.Lnowait%=:" :: "s"(skip), "n"(N) : "memory", "scc");
}
template <int N> __device__ __forceinline__ void waitvm_unless_n(unsigned skip, int n) {
    if (n == 0) waitvm_unless<N>(skip); else if (n == 1) waitvm_unless<N + 1>(skip); else if (n == 2) waitvm_unless<N + 2>(skip);
    else if (n == 3) waitvm_unless<N + 3>(skip); else if (n == 4) waitvm_unless<N + 4>(skip);
    else if (n == 5) waitvm_unless<N + 5>(skip); else waitvm_unless<N + 6>(skip);
}

__global__ void __launch_bounds__(NTHREADS, 1) gemm_ag_kernel(Params p) {
    __shared__ __attribute__((aligned(16))) char smem[LDS_TOTAL];
    const int tid = threadIdx.x;
    const int lane = tid & 63;
    const int wv = __builtin_amdgcn_readfirstlane(tid >> 6);
    const unsigned g = blockIdx.x;
    const unsigned lds0 = (unsigned)(size_t)(LAS char*)smem;
    const bool send_wg = g < NSEND;
    char* const myws = p.ws[0];
    GAS unsigned* const tmo = (GAS unsigned*)(myws + OFF_TMO);
    STAMP(0);

    const unsigned xrb = g >> 3;
    const unsigned xrel = (g + (g >> 3)) & 7u;
    u32x4 xv[NCH];
    unsigned tr[16];
    const unsigned tjob = (g < NSEND && wv == 1) ? g + NSEND : (g >= NSEND && g < 2 * NSEND && wv == 0) ? g - NSEND + 256u : g + 256u * wv;
    const unsigned tu = unit_order(tjob, p.rank);
    GAS unsigned* const head = (GAS unsigned*)(myws + OFF_HEAD);

    if (wv == W_SEND) {
        const __amdgpu_buffer_rsrc_t rs = __builtin_amdgcn_make_buffer_rsrc((void*)p.x, 0, 4096 * KS, 0x00020000);
        const unsigned d = (p.rank + xrel) & 7u;
#pragma unroll
        for (int j = 0; j < NCH; ++j)
            xv[j] = __builtin_amdgcn_raw_buffer_load_b128(
                rs, (int)((d * MS + xrb * 16u + (lane & 15)) * KS + j * 64u + (lane >> 4) * 16u), 0, 0);
    } else if (wv == W_LOAD) {
        unsigned c = 0;
        if (lane == 0) c = __hip_atomic_load((GAS unsigned*)(myws + OFF_CNT), __ATOMIC_RELAXED, __HIP_MEMORY_SCOPE_AGENT);
        if (lane < NCH) *(LAS unsigned*)(smem + L_SEQ + lane * 4) = 0u;
        if (lane >= 1 && lane < 16) *(LAS unsigned*)(smem + L_SYNC + lane * 4) = 0u;
        if (lane == 0) *(LAS unsigned*)(smem + S_CALL) = c / NWG + 1u;
        asm volatile("s_waitcnt lgkmcnt(0)" ::: "memory");
    } else if (wv < 2) {
        const unsigned u = tu;
        const __amdgpu_buffer_rsrc_t rs = __builtin_amdgcn_make_buffer_rsrc((void*)p.w, 0, KS * NN, 0x00020000);
#pragma unroll
        for (int t = 0; t < 16; ++t)
            tr[t] = __builtin_amdgcn_raw_buffer_load_b32(
                rs, (int)(((u >> 6) * 64u + (lane >> 4) * 16u + t) * NN + (u & 63u) * 64u + (lane & 15) * 4u), 0, 0);
    }
    __builtin_amdgcn_s_barrier();
    asm volatile("" ::: "memory");
    const unsigned call = lds_ld_u(lds0 + S_CALL);
    const unsigned par = OFF_DATA + (call & 1u) * PAR_STRIDE;
    const unsigned ci = (g - NSEND) >> 3;
    const unsigned cp = (g & 7u) * 4u + ci / 7u;
    const unsigned rg = ci % 7u;
    const unsigned nrb = rg < 4u ? 5u : 4u;
    const unsigned rb0 = rg < 4u ? rg * 5u : 20u + (rg - 4u) * 4u;
    STAMP(1);

    if (wv == W_SEND) {
        __builtin_amdgcn_s_setprio(2);
        char* const xd = p.ws[xrel];
        const unsigned xoff = par + W_BYTES + (p.rank * 32u + xrb) * 8192u + lane * 16u;
#pragma unroll
        for (int j = 0; j < NCH; ++j) asm volatile("" : "+v"(xv[j]));
        STAMP(2);
#pragma unroll
        for (int j = 0; j < NCH; ++j) {
            st16_wt(xd, xoff + j * 1024u, xv[j]);
            if (j < NCH - 1) waitvm<XWIN>();
        }
        GAS unsigned* const fx = (GAS unsigned*)(xd + OFF_FLAGX + xrb * 128u + p.rank * 4u);
        if (!send_wg) {
            waitvm<0>();
            if (lane == 0) __hip_atomic_store(fx, call, __ATOMIC_RELAXED, __HIP_MEMORY_SCOPE_SYSTEM);
            STAMP(3);
            STAMP_DUMP();
            return;
        }
        STAMP(3);
        char* peer = p.ws[0];
#pragma unroll
        for (int t = 1; t < NR; ++t) peer = ((lane & 7) == t) ? p.ws[t] : peer;
        char* dst[NR - 1];
#pragma unroll
        for (int t = 0; t < NR - 1; ++t) dst[t] = p.ws[1 + (t + g) % 7u];
        const unsigned qbase = (call - 1u) * TICKETS - 2u * NSEND;
        unsigned jcur = g;
        unsigned jnxt = g + NSEND;
        const unsigned ring = (unsigned)__builtin_amdgcn_readfirstlane(lds0 + L_RING);
        const unsigned wbase = par + p.rank * (NUNIT * 4096u);
        const char* wloc = myws + wbase + lane * 16;
        GAS unsigned* fprev = fx;
        bool fact = lane == 0;
        lds_wait_ge(lds0 + S_RING, 2u);
        unsigned slot = 0;
#define JOB_UNIT(jb) unit_order((jb), p.rank)
#define JOB_FLAG(jb) ((GAS unsigned*)(peer + OFF_FLAGW + (JOB_UNIT(jb) * 8u + p.rank) * 4u))
#pragma unroll 1
        for (unsigned nth = 0;; ++nth) {
            const unsigned last = (unsigned)__builtin_amdgcn_readfirstlane(jnxt >= NJOB ? 1 : 0);
            const char* nsrc = wloc + JOB_UNIT(last ? jcur : jnxt) * 4096u;
            const unsigned nslot = ring + (slot ^ 1u) * 4096u;
            unsigned qn = 0;
            u32x4 f[4];
            lds_rd4(f, ring + slot * 4096u + lane * 16u);
            const unsigned woff = wbase + JOB_UNIT(jcur) * 4096u + lane * 16u;
            STAMP(4 + (jcur >> 6));
#pragma unroll
            for (int k = 0; k < 28; ++k) {
                st16_wt(dst[k % 7], woff + (k / 7) * 1024u, f[k / 7]);
                const int n = ((WIN - 1 >= k - WIN && WIN - 1 <= k - 1) ? 1 : 0) + ((LDN >= k - WIN && LDN <= k - 1) ? 5 : 0);
                waitvm_unless_n<WIN>(last, n);
                if (k == WIN - 1) {
                    waitvm<WIN>();
                    if (fact) __hip_atomic_store(fprev, call, __ATOMIC_RELAXED, __HIP_MEMORY_SCOPE_SYSTEM);
                }
                if (k == LDN && !last) {
                    if (nth == 1) lds_wait_ge(lds0 + S_TDONE, 1u);
                    if (lane == 0) ticket_issue(qn, head);
                    if (nth != 0) glds64_sys(nsrc, nslot);
                }
            }
            fprev = JOB_FLAG(jcur);
            fact = lane >= 1 && lane < NR;
            if (last) break;
            waitvm<(27 - LDN) + (WIN - 1 > LDN ? 1 : 0)>();
            asm volatile("" : "+v"(qn));
            jcur = jnxt;
            jnxt = (unsigned)__builtin_amdgcn_readfirstlane(qn) - qbase;
            slot ^= 1u;
        }
        STAMP(12);
        waitvm<0>();
        STAMP(13);
        if (lane >= 1 && lane < NR) __hip_atomic_store(fprev, call, __ATOMIC_RELAXED, __HIP_MEMORY_SCOPE_SYSTEM);
        STAMP_DUMP();
        return;
    }

    if (wv == W_LOAD) {
        __builtin_amdgcn_s_setprio(3);
        if (send_wg) {
            const char* ft = myws + OFF_FLAGT + lane * 16;
            unsigned long long t0 = __builtin_amdgcn_s_memrealtime();
            for (unsigned spins = 0;; ++spins) {
                u32x4 a, b;
                asm volatile("global_load_dwordx4 %0, %2, off sc0 sc1\n\tglobal_load_dwordx4 %1, %2, off offset:1024 sc0 sc1\n\t"
                             "s_waitcnt vmcnt(0)" : "=&v"(a), "=&v"(b) : "v"(ft) : "memory");
                bool ok = true;
#pragma unroll
                for (int e = 0; e < 4; ++e) ok = ok && (int)(a[e] - call) >= 0 && (int)(b[e] - call) >= 0;
                if (__all(ok)) break;
                if (give_up(t0, spins, tmo, call)) break;
                __builtin_amdgcn_s_sleep(2);
            }
            if (lane == 0) lds_st(lds0 + S_TDONE, 1u);
            STAMP(12);
            if (lane == 0)
                (void)__hip_atomic_fetch_add((GAS unsigned*)(myws + OFF_CNT), 1u, __ATOMIC_RELAXED, __HIP_MEMORY_SCOPE_AGENT);
            STAMP_DUMP();
            return;
        }
        const char* wsrc = myws + par + cp * 8192u + lane * 16;
        const unsigned lbase = (unsigned)__builtin_amdgcn_readfirstlane(lds0 + L_B);
        const unsigned nmfma = 2u * nrb;
        unsigned cs0 = 0, cs1 = 1;
        bool act0 = false, act1 = false;
        unsigned pub0 = 0, pub1 = 0, tmo_now = 0;
        unsigned pending = 0, issued = 0;
        bool counted = false;
        const unsigned long long t0 = __builtin_amdgcn_s_memrealtime();
        for (unsigned spins = 0;; ++spins) {
            if (!act0 && cs0 < NCH && (cs0 < 2 || lds_ld_u(lds0 + S_DONE + (cs0 - 2) * 4u) >= nmfma)) {
                act0 = true; pub0 = 0; pending |= 0xffffu; STAMP(16 + cs0);
            }
            if (!act1 && cs1 < NCH && (cs1 < 2 || lds_ld_u(lds0 + S_DONE + (cs1 - 2) * 4u) >= nmfma)) {
                act1 = true; pub1 = 0; pending |= 0xffff0000u; STAMP(16 + cs1);
            }
            unsigned ready = 0;
            if (pending) {
                const unsigned cl = ((lane & 16) ? cs1 : cs0) & 7u;
                const unsigned fo = OFF_FLAGW + ((cl * 64u + cp * 2u + (lane & 1)) * 8u + ((lane >> 1) & 7)) * 4u;
                unsigned v = __hip_atomic_load((GAS unsigned*)(myws + fo), __ATOMIC_RELAXED, __HIP_MEMORY_SCOPE_SYSTEM);
                ready = (unsigned)__ballot((int)(v - call) >= 0) & pending;
                if (!ready && give_up(t0, spins, tmo, call)) ready = pending;
            }
            waitvm<0>();
            unsigned both = issued & (issued >> 1) & 0x55555555u;
            both = (both | (both >> 1)) & 0x33333333u;
            both = (both | (both >> 2)) & 0x0f0f0f0fu;
            both = (both | (both >> 4)) & 0x00ff00ffu;
            const unsigned n0 = both & 0xffu, n1 = both >> 16;
            const unsigned badbit = (!pending && cs0 >= NCH - 2 && cs1 >= NCH - 1 && (act0 || cs0 >= NCH) && (act1 || cs1 >= NCH) &&
                                     tmo_now == call) ? 0x100u : 0u;
            if (act0 && n0 != pub0) {
                pub0 = n0;
                if (lane == 0) lds_st(lds0 + L_SEQ + cs0 * 4u, pub0 | badbit);
                if (pub0 == 0xffu) { STAMP(4 + cs0); act0 = false; cs0 += 2; issued &= 0xffff0000u; }
            }
            if (act1 && n1 != pub1) {
                pub1 = n1;
                if (lane == 0) lds_st(lds0 + L_SEQ + cs1 * 4u, pub1 | badbit);
                if (pub1 == 0xffu) { STAMP(4 + cs1); act1 = false; cs1 += 2; issued &= 0xffffu; }
            }
            if (cs0 >= NCH && cs1 >= NCH) break;
            if (!counted && cs0 >= 2) {
                counted = true;
                if (lane == 0)
                    (void)__hip_atomic_fetch_add((GAS unsigned*)(myws + OFF_CNT), 1u, __ATOMIC_RELAXED, __HIP_MEMORY_SCOPE_AGENT);
            }
            if (!ready) { if (cs1 < NCH - 1) __builtin_amdgcn_s_sleep(1); continue; }
            pending &= ~ready;
            issued |= ready;
            while (ready) {
                const unsigned b = __builtin_ctz(ready);
                ready &= ready - 1;
                const unsigned c = b >= 16 ? cs1 : cs0, q = (b >> 1) & 7u, h = b & 1u;
                glds64_sys(wsrc + (q * NCH + c) * (64u * 4096u) + h * 4096u, lbase + (c & 1u) * 65536u + q * 8192u + h * 4096u);
            }
            if (!pending && cs0 >= NCH - 2 && cs1 >= NCH - 1)
                tmo_now = (unsigned)__builtin_amdgcn_readfirstlane(__hip_atomic_load(tmo, __ATOMIC_RELAXED, __HIP_MEMORY_SCOPE_AGENT));
        }
        STAMP_DUMP();
        return;
    }

    if (wv < 2) {
        const unsigned u = tu;
        u32x4 f[4];
#pragma unroll
        for (int q = 0; q < 4; ++q) {
            unsigned t0 = __builtin_amdgcn_perm(tr[4 * q + 1], tr[4 * q + 0], 0x05010400u);
            unsigned t1 = __builtin_amdgcn_perm(tr[4 * q + 1], tr[4 * q + 0], 0x07030602u);
            unsigned t2 = __builtin_amdgcn_perm(tr[4 * q + 3], tr[4 * q + 2], 0x05010400u);
            unsigned t3 = __builtin_amdgcn_perm(tr[4 * q + 3], tr[4 * q + 2], 0x07030602u);
            f[0][q] = __builtin_amdgcn_perm(t2, t0, 0x05040100u);
            f[1][q] = __builtin_amdgcn_perm(t2, t0, 0x07060302u);
            f[2][q] = __builtin_amdgcn_perm(t3, t1, 0x05040100u);
            f[3][q] = __builtin_amdgcn_perm(t3, t1, 0x07060302u);
        }
        const unsigned woff = par + (p.rank * NUNIT + u) * 4096u + lane * 16u;
#pragma unroll
        for (int i = 0; i < 4; ++i) st16_wt(myws, woff + i * 1024u, f[i]);
        if (send_wg) {
#pragma unroll
            for (int i = 0; i < 4; ++i) *(LAS u32x4*)(smem + L_RING + wv * 4096 + i * 1024 + lane * 16) = f[i];
            asm volatile("s_waitcnt lgkmcnt(0)" ::: "memory");
            if (lane == 0) lds_add(lds0 + S_RING, 1u);
        }
        waitvm<0>();
        if (lane == 0) {
            __hip_atomic_store((GAS unsigned*)(myws + OFF_FLAGT + u * 4u), call, __ATOMIC_RELAXED, __HIP_MEMORY_SCOPE_SYSTEM);
            __hip_atomic_store((GAS unsigned*)(myws + OFF_FLAGW + (u * 8u + p.rank) * 4u), call, __ATOMIC_RELAXED, __HIP_MEMORY_SCOPE_SYSTEM);
        }
    }
    if (send_wg || (unsigned)wv >= 2u * nrb) { STAMP_DUMP(); return; }
    float scale = p.sx[0] * p.sw[0];
    const unsigned half = (unsigned)wv >= nrb ? 1u : 0u;
    const unsigned rb = rb0 + (unsigned)wv - half * nrb;
    const unsigned nj = cp * 2u + half;
    const __amdgpu_buffer_rsrc_t rsX =
        __builtin_amdgcn_make_buffer_rsrc((void*)(myws + par + W_BYTES), 0, (int)X_BYTES, 0x00020000);

    float c2 = scale * -1.4426950408889634f;
    char* const outp = (char*)p.out;
    unsigned ooff = ((rb * 16u + (lane >> 4) * 4u) * NN + nj * 64u + (lane & 15) * 4u) * 4u;
    asm volatile("" : "+v"(scale), "+v"(c2), "+v"(ooff) : "s"(outp));

    i32x4 acc[4] = {};
    unsigned bad = 0;
    int npass = 2;
    asm volatile("" : "+s"(npass));
#pragma unroll 1
    for (int pass = 0; pass < npass; ++pass) {
    if (pass != 0) {
        wait_flags((GAS unsigned*)(myws + OFF_FLAGX + rb * 128u + (lane & 7) * 4u), call, tmo, call);

        STAMP(2);
#pragma unroll 1
        for (int j = 0; j < NCH; ++j) {
            u32x32 a;
#pragma unroll
            for (int q = 0; q < NR; ++q) {
                const u32x4 t = __builtin_amdgcn_raw_buffer_load_b128(rsX, (int)(((q * 32u + rb) * 8u + j) * 1024u + lane * 16u), 0, 17);
                a[4 * q] = t[0]; a[4 * q + 1] = t[1]; a[4 * q + 2] = t[2]; a[4 * q + 3] = t[3];
            }
            const char* lb = smem + L_B + (j & 1) * 65536 + half * 4096 + lane * 16;
            unsigned seen = 0, m = 0;
#pragma unroll 1
            for (int left = NR; left > 0;) {
                if (m == 0u) {
                    const unsigned word = lds_ld_u(lds0 + L_SEQ + j * 4u);
                    bad |= word >> 8;
                    m = word & 0xffu & ~seen;
                    if (m == 0u) { __builtin_amdgcn_s_sleep(1); continue; }
                    if (seen == 0) STAMP(4 + j * 3);
                    seen |= m;
                }
                const unsigned q = __builtin_ctz(m);
                m &= m - 1;
                --left;
                i32x4 b[4];
#pragma unroll
                for (int i = 0; i < 4; ++i) b[i] = *(const LAS i32x4*)(lb + q * 8192u + i * 1024);
                const u32x4 av = {a[4 * q], a[4 * q + 1], a[4 * q + 2], a[4 * q + 3]};
#pragma unroll
                for (int i = 0; i < 4; ++i)
                    acc[i] = __builtin_amdgcn_mfma_i32_16x16x64_i8(__builtin_bit_cast(i32x4, av), b[i], acc[i], 0, 0, 0);
            }
            STAMP(5 + j * 3);
            if (j < NCH - 2 && lane == 0) lds_add(lds0 + S_DONE + j * 4u, 1u);
        }
    }
    STAMP(28);
    const __amdgpu_buffer_rsrc_t rsO = __builtin_amdgcn_make_buffer_rsrc((void*)outp, 0, pass != 0 ? MS * NN * 4 : 0, 0x00020000);
    if (__builtin_expect(bad != 0u, 0)) {
        const u32x4 nan4 = {0x7fc00000u, 0x7fc00000u, 0x7fc00000u, 0x7fc00000u};
#pragma unroll
        for (int r = 0; r < 4; ++r) __builtin_amdgcn_raw_buffer_store_b128(nan4, rsO, (int)(ooff + r * (NN * 4)), 0, 17);
    } else {
        constexpr float TMAX = 60.0f * 1.4426950408889634f;
#pragma unroll
        for (int r = 0; r < 4; ++r) {
            u32x4 o;
#pragma unroll
            for (int i = 0; i < 4; i += 2) {
                const f32x2 a = {(float)acc[i][r], (float)acc[i + 1][r]};
                const f32x2 y = a * scale;
                f32x2 t = a * c2;
                t[0] = __builtin_amdgcn_exp2f(__builtin_amdgcn_fmed3f(t[0], -TMAX, TMAX));
                t[1] = __builtin_amdgcn_exp2f(__builtin_amdgcn_fmed3f(t[1], -TMAX, TMAX));
                t = t + 1.0f;
                t[0] = __builtin_amdgcn_rcpf(t[0]);
                t[1] = __builtin_amdgcn_rcpf(t[1]);
                const f32x2 v = y * t;
                const float v0 = v[0], v1 = v[1];
                o[i] = __builtin_bit_cast(unsigned, v0);
                o[i + 1] = __builtin_bit_cast(unsigned, v1);
            }
            __builtin_amdgcn_raw_buffer_store_b128(o, rsO, (int)(ooff + r * (NN * 4)), 0, 17);
        }
    }
    }
    STAMP(29);
}

}

extern "C" void kernel_launch(int rank, int nranks, ncclComm_t comm,
                              hipStream_t stream,
                              void* const* d_in, const int* in_sizes, int n_in,
                              void* d_out, int out_size,
                              const int* in_dtypes, int out_dtype,
                              void* const* d_out_all,
                              void* const* d_ws_all, size_t ws_bytes) {
    (void)comm; (void)in_dtypes; (void)out_dtype; (void)d_out_all;
    if (nranks != NR || (unsigned)rank >= (unsigned)NR || n_in != 4 || ws_bytes < WS_NEEDED ||
        in_sizes[0] != 4096 * KS || in_sizes[1] != KS * NN || out_size != MS * NN) {
        fprintf(stderr, "kernel_launch: unexpected problem shape\n");
        return;
    }
    Params p;
    p.x = (const int8_t*)d_in[0];
    p.w = (const int8_t*)d_in[1];
    p.sx = (const float*)d_in[2];
    p.sw = (const float*)d_in[3];
    p.out = (float*)d_out;
    for (int t = 0; t < NR; ++t) p.ws[t] = (char*)d_ws_all[(rank + t) % NR];
    p.rank = rank;
    p.pad = 0;
    hipLaunchKernelGGL(gemm_ag_kernel, dim3(NWG), dim3(NTHREADS), 0, stream, p);
    HIP_CHECK(hipGetLastError());
}
```

```cpp
#include <hip/hip_runtime.h>
#include <rccl/rccl.h>
#include <cstdio>
#include <cstdint>

#define HIP_CHECK(expr) do { hipError_t _e = (expr); if (_e != hipSuccess) { \
    fprintf(stderr, "HIP error %s at %s:%d\n", hipGetErrorString(_e), __FILE__, __LINE__); \
    return; } } while (0)

namespace {

#define GAS __attribute__((address_space(1)))
#define LAS __attribute__((address_space(3)))
typedef unsigned u32x4 __attribute__((ext_vector_type(4)));
typedef int i32x4 __attribute__((ext_vector_type(4)));
typedef float f32x4 __attribute__((ext_vector_type(4)));
typedef float f32x2 __attribute__((ext_vector_type(2)));
typedef unsigned u32x32 __attribute__((ext_vector_type(32)));

constexpr int NR = 8;
constexpr int MS = 512;
constexpr int KS = 512;
constexpr int NN = 4096;
constexpr int NCH = 8;
constexpr int NWG = 256;
constexpr int NUNIT = NCH * 64;
constexpr int NCW = 10;
constexpr int W_LOAD = 10, W_SEND = 11;
constexpr int NWAVE = 12;
constexpr int NTHREADS = NWAVE * 64;
constexpr int NSEND = 32;
#define WIN 10
#define LDN 12
#define TDELAY 8
#define XWIN 1
constexpr unsigned NJOB = NUNIT;
static_assert(WIN >= 2 && WIN <= 14 && LDN >= 2 && LDN + WIN <= 27 && 2 * NSEND <= 64, "sender window");
constexpr unsigned TICKETS = NJOB - NSEND;

constexpr unsigned OFF_CNT = 0;
constexpr unsigned OFF_TMO = 256;
constexpr unsigned OFF_HEAD = 512;
constexpr unsigned OFF_FLAGT = 4096;
constexpr unsigned OFF_FLAGX = 8192;
constexpr unsigned OFF_FLAGW = 16384;
constexpr unsigned OFF_DATA = 1u << 20;
constexpr unsigned W_BYTES = 16u << 20;
constexpr unsigned X_BYTES = 2u << 20;
constexpr unsigned PAR_STRIDE = W_BYTES + X_BYTES;
constexpr size_t WS_NEEDED = (size_t)OFF_DATA + 2 * (size_t)PAR_STRIDE;

constexpr unsigned L_B = 0;
constexpr unsigned L_RING = 0;
constexpr unsigned L_SEQ = 131072;
constexpr unsigned L_SYNC = 131328;
constexpr unsigned S_CALL = L_SYNC, S_RING = L_SYNC + 8, S_TDONE = L_SYNC + 12;
constexpr unsigned S_DONE = L_SYNC + 32;
constexpr unsigned LDS_BYTES = L_SYNC + 256;

#define DEADLINE 100000000ull
constexpr unsigned long long DEADLINE_TICKS = DEADLINE;

struct Params {
    const int8_t* x;
    const int8_t* w;
    const float* sx;
    const float* sw;
    float* out;
    char* ws[NR];
    int rank;
    int pad;
};

#define STAMP(i) do {} while (0)
#define STAMP_DUMP() do {} while (0)
constexpr unsigned LDS_TOTAL = LDS_BYTES;

template <int N> __device__ __forceinline__ void waitvm() { asm volatile("s_waitcnt vmcnt(%0)" :: "n"(N) : "memory"); }
__device__ __forceinline__ void st16_wt(char* base, unsigned off, u32x4 v) {
    asm volatile("global_store_dwordx4 %0, %1, %2 sc0 sc1\n\ts_nop 1" :: "v"(off), "v"(v), "s"(base) : "memory");
}
__device__ __forceinline__ void glds16_sys(const void* gsrc, unsigned lds_dst) {
    unsigned keep;
    asm volatile("s_mov_b32 %0, m0\n\ts_mov_b32 m0, %2\n\ts_nop 0\n\tglobal_load_lds_dwordx4 %1, off sc0 sc1\n\ts_mov_b32 m0, %0"
                 : "=&s"(keep) : "v"(gsrc), "s"(lds_dst) : "memory");
}
__device__ __forceinline__ void glds64_sys(const void* gsrc, unsigned lds_dst) {
    unsigned keep;
    asm volatile("s_mov_b32 %0, m0\n\ts_mov_b32 m0, %2\n\ts_nop 0\n\t"
                 "global_load_lds_dwordx4 %1, off sc0 sc1\n\tglobal_load_lds_dwordx4 %1, off offset:1024 sc0 sc1\n\t"
                 "global_load_lds_dwordx4 %1, off offset:2048 sc0 sc1\n\tglobal_load_lds_dwordx4 %1, off offset:3072 sc0 sc1\n\t"
                 "s_mov_b32 m0, %0"
                 : "=&s"(keep) : "v"(gsrc), "s"(lds_dst) : "memory");
}
__device__ __forceinline__ void lds_rd4(u32x4 (&f)[4], unsigned addr) {
    asm volatile("ds_read_b128 %0, %4\n\tds_read_b128 %1, %4 offset:1024\n\tds_read_b128 %2, %4 offset:2048\n\t"
                 "ds_read_b128 %3, %4 offset:3072\n\ts_waitcnt lgkmcnt(0)"
                 : "=&v"(f[0]), "=&v"(f[1]), "=&v"(f[2]), "=&v"(f[3]) : "v"(addr) : "memory");
}
__device__ __forceinline__ u32x4 lds_rd1(unsigned addr) {
    u32x4 f;
    asm volatile("ds_read_b128 %0, %1\n\ts_waitcnt lgkmcnt(0)" : "=v"(f) : "v"(addr) : "memory");
    return f;
}
__device__ __forceinline__ void ticket_issue(unsigned& q, GAS unsigned* head) {
    unsigned zero = 0u, one = 1u;
    asm volatile("global_atomic_add %0, %1, %2, %3 sc0" : "=v"(q) : "v"(zero), "v"(one), "s"(head) : "memory");
}
__device__ __forceinline__ void lds_add(unsigned addr, unsigned v) {
    asm volatile("ds_add_u32 %0, %1" :: "v"(addr), "v"(v) : "memory");
}
__device__ __forceinline__ void lds_st(unsigned addr, unsigned v) {
    asm volatile("ds_write_b32 %0, %1" :: "v"(addr), "v"(v) : "memory");
}
__device__ __forceinline__ unsigned lds_ld(unsigned addr) {
    unsigned r;
    asm volatile("ds_read_b32 %0, %1\n\ts_waitcnt lgkmcnt(0)" : "=v"(r) : "v"(addr) : "memory");
    return r;
}
__device__ __forceinline__ unsigned lds_ld_u(unsigned addr) { return (unsigned)__builtin_amdgcn_readfirstlane(lds_ld(addr)); }
__device__ __forceinline__ void lds_wait_ge(unsigned addr, unsigned want) {
    while (lds_ld_u(addr) < want) __builtin_amdgcn_s_sleep(1);
}

__device__ __forceinline__ bool give_up(unsigned long long t0, unsigned spins, GAS unsigned* tmo, unsigned call) {
    bool g = __builtin_amdgcn_s_memrealtime() - t0 > DEADLINE_TICKS;
    if ((spins & 15u) == 15u)
        g |= (unsigned)__builtin_amdgcn_readfirstlane(
                 __hip_atomic_load(tmo, __ATOMIC_RELAXED, __HIP_MEMORY_SCOPE_AGENT)) == call;
    if (g) __hip_atomic_store(tmo, call, __ATOMIC_RELAXED, __HIP_MEMORY_SCOPE_AGENT);
    return g;
}
__device__ __forceinline__ void wait_flags(GAS unsigned* flag, unsigned want, GAS unsigned* tmo, unsigned call) {
    unsigned long long t0 = __builtin_amdgcn_s_memrealtime();
    for (unsigned spins = 0;; ++spins) {
        unsigned v = __hip_atomic_load(flag, __ATOMIC_RELAXED, __HIP_MEMORY_SCOPE_SYSTEM);
        if (__all((int)(v - want) >= 0)) break;
        if (give_up(t0, spins, tmo, call)) break;
        __builtin_amdgcn_s_sleep(4);
    }
}

__device__ __forceinline__ unsigned unit_order(unsigned u, unsigned rank) {
    return (u & ~63u) | ((u + 8u * rank) & 63u);
}

template <int N> __device__ __forceinline__ void waitvm_unless(unsigned skip) {
    asm volatile("s_cmp_lg_u32 %0, 0\n\ts_cbranch_scc1 .Lnowait%=\n\ts_waitcnt vmcnt(%1)\n.Lnowait%=:" :: "s"(skip), "n"(N) : "memory", "scc");
}
template <int N> __device__ __forceinline__ void waitvm_unless_n(unsigned skip, int n) {
    if (n == 0) waitvm_unless<N>(skip); else if (n == 1) waitvm_unless<N + 1>(skip); else if (n == 2) waitvm_unless<N + 2>(skip);
    else if (n == 3) waitvm_unless<N + 3>(skip); else if (n == 4) waitvm_unless<N + 4>(skip);
    else if (n == 5) waitvm_unless<N + 5>(skip); else waitvm_unless<N + 6>(skip);
}

__global__ void __launch_bounds__(NTHREADS, 1) gemm_ag_kernel(Params p) {
    __shared__ __attribute__((aligned(16))) char smem[LDS_TOTAL];
    const int tid = threadIdx.x;
    const int lane = tid & 63;
    const int wv = __builtin_amdgcn_readfirstlane(tid >> 6);
    const unsigned g = blockIdx.x;
    const unsigned lds0 = (unsigned)(size_t)(LAS char*)smem;
    const bool send_wg = g < NSEND;
    char* const myws = p.ws[0];
    GAS unsigned* const tmo = (GAS unsigned*)(myws + OFF_TMO);
    STAMP(0);

    const unsigned xrb = g >> 3;
    const unsigned xrel = (g + (g >> 3)) & 7u;
    u32x4 xv[NCH];
    unsigned tr[16];
    const unsigned tu = unit_order(2u * g + wv, p.rank);
    GAS unsigned* const head = (GAS unsigned*)(myws + OFF_HEAD);

    if (wv == W_SEND) {
        unsigned cs;
        {
            unsigned zero = 0u;
            asm volatile("global_load_dword %0, %1, %2 sc1" : "=v"(cs) : "v"(zero), "s"(myws + OFF_CNT) : "memory");
        }
        const __amdgpu_buffer_rsrc_t rs = __builtin_amdgcn_make_buffer_rsrc((void*)p.x, 0, 4096 * KS, 0x00020000);
        const unsigned d = (p.rank + xrel) & 7u;
#pragma unroll
        for (int j = 0; j < NCH; ++j)
            xv[j] = __builtin_amdgcn_raw_buffer_load_b128(
                rs, (int)((d * MS + xrb * 16u + (lane & 15)) * KS + j * 64u + (lane >> 4) * 16u), 0, 0);
        asm volatile("s_waitcnt vmcnt(0)" : "+v"(cs) :: "memory");
#pragma unroll
        for (int j = 0; j < NCH; ++j) asm volatile("" : "+v"(xv[j]));
        const unsigned scall = (unsigned)__builtin_amdgcn_readfirstlane(cs) / NWG + 1u;
        const unsigned xoff = OFF_DATA + (scall & 1u) * PAR_STRIDE + W_BYTES + (p.rank * 32u + xrb) * 8192u + lane * 16u;
#pragma unroll
        for (int j = 0; j <= XWIN; ++j) st16_wt(p.ws[xrel], xoff + j * 1024u, xv[j]);
    } else if (wv == W_LOAD) {
        unsigned c = 0;
        if (lane == 0) c = __hip_atomic_load((GAS unsigned*)(myws + OFF_CNT), __ATOMIC_RELAXED, __HIP_MEMORY_SCOPE_AGENT);
        if (lane < NCH) *(LAS unsigned*)(smem + L_SEQ + lane * 4) = 0u;
        if (lane >= 1 && lane < 16) *(LAS unsigned*)(smem + L_SYNC + lane * 4) = 0u;
        if (lane == 0) *(LAS unsigned*)(smem + S_CALL) = c / NWG + 1u;
        asm volatile("s_waitcnt lgkmcnt(0)" ::: "memory");
    } else if (wv < 2) {
        if (TDELAY) __builtin_amdgcn_s_sleep(TDELAY);
        const unsigned u = tu;
        const __amdgpu_buffer_rsrc_t rs = __builtin_amdgcn_make_buffer_rsrc((void*)p.w, 0, KS * NN, 0x00020000);
#pragma unroll
        for (int t = 0; t < 16; ++t)
            tr[t] = __builtin_amdgcn_raw_buffer_load_b32(
                rs, (int)(((u >> 6) * 64u + (lane >> 4) * 16u + t) * NN + (u & 63u) * 64u + (lane & 15) * 4u), 0, 0);
    }
    __builtin_amdgcn_s_barrier();
    asm volatile("" ::: "memory");
    const unsigned call = lds_ld_u(lds0 + S_CALL);
    const unsigned par = OFF_DATA + (call & 1u) * PAR_STRIDE;
    const unsigned ci = (g - NSEND) >> 3;
    const unsigned cp = (g & 7u) * 4u + ci / 7u;
    const unsigned rg = ci % 7u;
    const unsigned nrb = rg < 4u ? 5u : 4u;
    const unsigned rb0 = rg < 4u ? rg * 5u : 20u + (rg - 4u) * 4u;
    STAMP(1);

    if (wv == W_SEND) {
        __builtin_amdgcn_s_setprio(2);
        char* const xd = p.ws[xrel];
        const unsigned xoff = par + W_BYTES + (p.rank * 32u + xrb) * 8192u + lane * 16u;
        STAMP(2);
#pragma unroll
        for (int j = XWIN + 1; j < NCH; ++j) {
            waitvm<XWIN>();
            st16_wt(xd, xoff + j * 1024u, xv[j]);
        }
        GAS unsigned* const fx = (GAS unsigned*)(xd + OFF_FLAGX + xrb * 128u + p.rank * 4u);
        if (!send_wg) {
            waitvm<0>();
            if (lane == 0) __hip_atomic_store(fx, call, __ATOMIC_RELAXED, __HIP_MEMORY_SCOPE_SYSTEM);
            STAMP(3);
            STAMP_DUMP();
            return;
        }
        STAMP(3);
        char* peer = p.ws[0];
#pragma unroll
        for (int t = 1; t < NR; ++t) peer = ((lane & 7) == t) ? p.ws[t] : peer;
        char* dst[NR - 1];
#pragma unroll
        for (int t = 0; t < NR - 1; ++t) dst[t] = p.ws[1 + (t + g) % 7u];
        const unsigned qbase = (call - 1u) * TICKETS - 2u * NSEND;
        unsigned jcur = 2u * g;
        unsigned jnxt = 2u * g + 1u;
        const unsigned ring = (unsigned)__builtin_amdgcn_readfirstlane(lds0 + L_RING);
        const unsigned wbase = par + p.rank * (NUNIT * 4096u);
        const char* wloc = myws + wbase + lane * 16;
        GAS unsigned* fprev = fx;
        bool fact = lane == 0;
        lds_wait_ge(lds0 + S_RING, 2u);
        unsigned slot = 0;
#define JOB_UNIT(jb) unit_order((jb), p.rank)
#define JOB_FLAG(jb) ((GAS unsigned*)(peer + OFF_FLAGW + (JOB_UNIT(jb) * 8u + p.rank) * 4u))
#pragma unroll 1
        for (unsigned nth = 0;; ++nth) {
            const unsigned last = (unsigned)__builtin_amdgcn_readfirstlane(jnxt >= NJOB ? 1 : 0);
            const char* nsrc = wloc + JOB_UNIT(last ? jcur : jnxt) * 4096u;
            const unsigned nslot = ring + (slot ^ 1u) * 4096u;
            unsigned qn = 0;
            u32x4 f[4];
            lds_rd4(f, ring + slot * 4096u + lane * 16u);
            const unsigned woff = wbase + JOB_UNIT(jcur) * 4096u + lane * 16u;
            STAMP(4 + (jcur >> 6));
#pragma unroll
            for (int k = 0; k < 28; ++k) {
                st16_wt(dst[k % 7], woff + (k / 7) * 1024u, f[k / 7]);
                const int n = ((WIN - 1 >= k - WIN && WIN - 1 <= k - 1) ? 1 : 0) + ((LDN >= k - WIN && LDN <= k - 1) ? 5 : 0);
                waitvm_unless_n<WIN>(last, n);
                if (k == WIN - 1) {
                    waitvm<WIN>();
                    if (fact) __hip_atomic_store(fprev, call, __ATOMIC_RELAXED, __HIP_MEMORY_SCOPE_SYSTEM);
                }
                if (k == LDN && !last) {
                    if (nth == 1) lds_wait_ge(lds0 + S_TDONE, 1u);
                    if (lane == 0) ticket_issue(qn, head);
                    if (nth != 0) glds64_sys(nsrc, nslot);
                }
            }
            fprev = JOB_FLAG(jcur);
            fact = lane >= 1 && lane < NR;
            if (last) break;
            waitvm<(27 - LDN) + (WIN - 1 > LDN ? 1 : 0)>();
            asm volatile("" : "+v"(qn));
            jcur = jnxt;
            jnxt = (unsigned)__builtin_amdgcn_readfirstlane(qn) - qbase;
            slot ^= 1u;
        }
        STAMP(12);
        waitvm<0>();
        STAMP(13);
        if (lane >= 1 && lane < NR) __hip_atomic_store(fprev, call, __ATOMIC_RELAXED, __HIP_MEMORY_SCOPE_SYSTEM);
        STAMP_DUMP();
        return;
    }

    if (wv == W_LOAD) {
        __builtin_amdgcn_s_setprio(3);
        if (send_wg) {
            const char* ft = myws + OFF_FLAGT + lane * 16;
            unsigned long long t0 = __builtin_amdgcn_s_memrealtime();
            for (unsigned spins = 0;; ++spins) {
                u32x4 a, b;
                asm volatile("global_load_dwordx4 %0, %2, off sc0 sc1\n\tglobal_load_dwordx4 %1, %2, off offset:1024 sc0 sc1\n\t"
                             "s_waitcnt vmcnt(0)" : "=&v"(a), "=&v"(b) : "v"(ft) : "memory");
                bool ok = true;
#pragma unroll
                for (int e = 0; e < 4; ++e) ok = ok && (int)(a[e] - call) >= 0 && (int)(b[e] - call) >= 0;
                if (__all(ok)) break;
                if (give_up(t0, spins, tmo, call)) break;
                __builtin_amdgcn_s_sleep(2);
            }
            if (lane == 0) lds_st(lds0 + S_TDONE, 1u);
            STAMP(12);
            if (lane == 0)
                (void)__hip_atomic_fetch_add((GAS unsigned*)(myws + OFF_CNT), 1u, __ATOMIC_RELAXED, __HIP_MEMORY_SCOPE_AGENT);
            STAMP_DUMP();
            return;
        }
        const char* wsrc = myws + par + cp * 8192u + lane * 16;
        const unsigned lbase = (unsigned)__builtin_amdgcn_readfirstlane(lds0 + L_B);
        const unsigned nmfma = 2u * nrb;
        unsigned cs0 = 0, cs1 = 1;
        bool act0 = false, act1 = false;
        unsigned pub0 = 0, pub1 = 0, tmo_now = 0;
        unsigned pending = 0, issued = 0;
        bool counted = false;
        const unsigned long long t0 = __builtin_amdgcn_s_memrealtime();
        for (unsigned spins = 0;; ++spins) {
            if (!act0 && cs0 < NCH && (cs0 < 2 || lds_ld_u(lds0 + S_DONE + (cs0 - 2) * 4u) >= nmfma)) {
                act0 = true; pub0 = 0; pending |= 0xffffu; STAMP(16 + cs0);
            }
            if (!act1 && cs1 < NCH && (cs1 < 2 || lds_ld_u(lds0 + S_DONE + (cs1 - 2) * 4u) >= nmfma)) {
                act1 = true; pub1 = 0; pending |= 0xffff0000u; STAMP(16 + cs1);
            }
            unsigned ready = 0;
            if (pending) {
                const unsigned cl = ((lane & 16) ? cs1 : cs0) & 7u;
                const unsigned fo = OFF_FLAGW + ((cl * 64u + cp * 2u + (lane & 1)) * 8u + ((lane >> 1) & 7)) * 4u;
                unsigned v = __hip_atomic_load((GAS unsigned*)(myws + fo), __ATOMIC_RELAXED, __HIP_MEMORY_SCOPE_SYSTEM);
                ready = (unsigned)__ballot((int)(v - call) >= 0) & pending;
                if (!ready && give_up(t0, spins, tmo, call)) ready = pending;
            }
            waitvm<0>();
            unsigned both = issued & (issued >> 1) & 0x55555555u;
            both = (both | (both >> 1)) & 0x33333333u;
            both = (both | (both >> 2)) & 0x0f0f0f0fu;
            both = (both | (both >> 4)) & 0x00ff00ffu;
            const unsigned n0 = both & 0xffu, n1 = both >> 16;
            const unsigned badbit = (!pending && cs0 >= NCH - 2 && cs1 >= NCH - 1 && (act0 || cs0 >= NCH) && (act1 || cs1 >= NCH) &&
                                     tmo_now == call) ? 0x100u : 0u;
            if (act0 && n0 != pub0) {
                pub0 = n0;
                if (lane == 0) lds_st(lds0 + L_SEQ + cs0 * 4u, pub0 | badbit);
                if (pub0 == 0xffu) { STAMP(4 + cs0); act0 = false; cs0 += 2; issued &= 0xffff0000u; }
            }
            if (act1 && n1 != pub1) {
                pub1 = n1;
                if (lane == 0) lds_st(lds0 + L_SEQ + cs1 * 4u, pub1 | badbit);
                if (pub1 == 0xffu) { STAMP(4 + cs1); act1 = false; cs1 += 2; issued &= 0xffffu; }
            }
            if (cs0 >= NCH && cs1 >= NCH) break;
            if (!counted && cs0 >= 2) {
                counted = true;
                if (lane == 0)
                    (void)__hip_atomic_fetch_add((GAS unsigned*)(myws + OFF_CNT), 1u, __ATOMIC_RELAXED, __HIP_MEMORY_SCOPE_AGENT);
            }
            if (!ready) { if (cs1 < NCH - 1) __builtin_amdgcn_s_sleep(1); continue; }
            pending &= ~ready;
            issued |= ready;
            while (ready) {
                const unsigned b = __builtin_ctz(ready);
                ready &= ready - 1;
                const unsigned c = b >= 16 ? cs1 : cs0, q = (b >> 1) & 7u, h = b & 1u;
                glds64_sys(wsrc + (q * NCH + c) * (64u * 4096u) + h * 4096u, lbase + (c & 1u) * 65536u + q * 8192u + h * 4096u);
            }
            if (!pending && cs0 >= NCH - 2 && cs1 >= NCH - 1)
                tmo_now = (unsigned)__builtin_amdgcn_readfirstlane(__hip_atomic_load(tmo, __ATOMIC_RELAXED, __HIP_MEMORY_SCOPE_AGENT));
        }
        STAMP_DUMP();
        return;
    }

    if (wv < 2) {
        const unsigned u = tu;
        u32x4 f[4];
#pragma unroll
        for (int q = 0; q < 4; ++q) {
            unsigned t0 = __builtin_amdgcn_perm(tr[4 * q + 1], tr[4 * q + 0], 0x05010400u);
            unsigned t1 = __builtin_amdgcn_perm(tr[4 * q + 1], tr[4 * q + 0], 0x07030602u);
            unsigned t2 = __builtin_amdgcn_perm(tr[4 * q + 3], tr[4 * q + 2], 0x05010400u);
            unsigned t3 = __builtin_amdgcn_perm(tr[4 * q + 3], tr[4 * q + 2], 0x07030602u);
            f[0][q] = __builtin_amdgcn_perm(t2, t0, 0x05040100u);
            f[1][q] = __builtin_amdgcn_perm(t2, t0, 0x07060302u);
            f[2][q] = __builtin_amdgcn_perm(t3, t1, 0x05040100u);
            f[3][q] = __builtin_amdgcn_perm(t3, t1, 0x07060302u);
        }
        const unsigned woff = par + (p.rank * NUNIT + u) * 4096u + lane * 16u;
#pragma unroll
        for (int i = 0; i < 4; ++i) st16_wt(myws, woff + i * 1024u, f[i]);
        if (send_wg) {
#pragma unroll
            for (int i = 0; i < 4; ++i) *(LAS u32x4*)(smem + L_RING + wv * 4096 + i * 1024 + lane * 16) = f[i];
            asm volatile("s_waitcnt lgkmcnt(0)" ::: "memory");
            if (lane == 0) lds_add(lds0 + S_RING, 1u);
        }
        waitvm<0>();
        if (lane == 0) {
            __hip_atomic_store((GAS unsigned*)(myws + OFF_FLAGT + u * 4u), call, __ATOMIC_RELAXED, __HIP_MEMORY_SCOPE_SYSTEM);
            __hip_atomic_store((GAS unsigned*)(myws + OFF_FLAGW + (u * 8u + p.rank) * 4u), call, __ATOMIC_RELAXED, __HIP_MEMORY_SCOPE_SYSTEM);
        }
    }
    if (send_wg || (unsigned)wv >= 2u * nrb) { STAMP_DUMP(); return; }
    float scale = p.sx[0] * p.sw[0];
    const unsigned half = (unsigned)wv >= nrb ? 1u : 0u;
    const unsigned rb = rb0 + (unsigned)wv - half * nrb;
    const unsigned nj = cp * 2u + half;
    const __amdgpu_buffer_rsrc_t rsX =
        __builtin_amdgcn_make_buffer_rsrc((void*)(myws + par + W_BYTES), 0, (int)X_BYTES, 0x00020000);

    float c2 = scale * -1.4426950408889634f;
    char* const outp = (char*)p.out;
    unsigned ooff = ((rb * 16u + (lane >> 4) * 4u) * NN + nj * 64u + (lane & 15) * 4u) * 4u;
    asm volatile("" : "+v"(scale), "+v"(c2), "+v"(ooff) : "s"(outp));

    i32x4 acc[4] = {};
    unsigned bad = 0;
    int npass = 2;
    asm volatile("" : "+s"(npass));
#pragma unroll 1
    for (int pass = 0; pass < npass; ++pass) {
    if (pass != 0) {
        wait_flags((GAS unsigned*)(myws + OFF_FLAGX + rb * 128u + (lane & 7) * 4u), call, tmo, call);

        STAMP(2);
#pragma unroll 1
        for (int j = 0; j < NCH; ++j) {
            u32x32 a;
#pragma unroll
            for (int q = 0; q < NR; ++q) {
                const u32x4 t = __builtin_amdgcn_raw_buffer_load_b128(rsX, (int)(((q * 32u + rb) * 8u + j) * 1024u + lane * 16u), 0, 17);
                a[4 * q] = t[0]; a[4 * q + 1] = t[1]; a[4 * q + 2] = t[2]; a[4 * q + 3] = t[3];
            }
            const char* lb = smem + L_B + (j & 1) * 65536 + half * 4096 + lane * 16;
            unsigned seen = 0, m = 0;
#pragma unroll 1
            for (int left = NR; left > 0;) {
                if (m == 0u) {
                    const unsigned word = lds_ld_u(lds0 + L_SEQ + j * 4u);
                    bad |= word >> 8;
                    m = word & 0xffu & ~seen;
                    if (m == 0u) { __builtin_amdgcn_s_sleep(1); continue; }
                    if (seen == 0) STAMP(4 + j * 3);
                    seen |= m;
                }
                const unsigned q = __builtin_ctz(m);
                m &= m - 1;
                --left;
                i32x4 b[4];
#pragma unroll
                for (int i = 0; i < 4; ++i) b[i] = *(const LAS i32x4*)(lb + q * 8192u + i * 1024);
                const u32x4 av = {a[4 * q], a[4 * q + 1], a[4 * q + 2], a[4 * q + 3]};
#pragma unroll
                for (int i = 0; i < 4; ++i)
                    acc[i] = __builtin_amdgcn_mfma_i32_16x16x64_i8(__builtin_bit_cast(i32x4, av), b[i], acc[i], 0, 0, 0);
            }
            STAMP(5 + j * 3);
            if (j < NCH - 2 && lane == 0) lds_add(lds0 + S_DONE + j * 4u, 1u);
        }
    }
    STAMP(28);
    const __amdgpu_buffer_rsrc_t rsO = __builtin_amdgcn_make_buffer_rsrc((void*)outp, 0, pass != 0 ? MS * NN * 4 : 0, 0x00020000);
    if (__builtin_expect(bad != 0u, 0)) {
        const u32x4 nan4 = {0x7fc00000u, 0x7fc00000u, 0x7fc00000u, 0x7fc00000u};
#pragma unroll
        for (int r = 0; r < 4; ++r) __builtin_amdgcn_raw_buffer_store_b128(nan4, rsO, (int)(ooff + r * (NN * 4)), 0, 17);
    } else {
        constexpr float TMAX = 60.0f * 1.4426950408889634f;
#pragma unroll
        for (int r = 0; r < 4; ++r) {
            u32x4 o;
#pragma unroll
            for (int i = 0; i < 4; i += 2) {
                const f32x2 a = {(float)acc[i][r], (float)acc[i + 1][r]};
                const f32x2 y = a * scale;
                f32x2 t = a * c2;
                t[0] = __builtin_amdgcn_exp2f(__builtin_amdgcn_fmed3f(t[0], -TMAX, TMAX));
                t[1] = __builtin_amdgcn_exp2f(__builtin_amdgcn_fmed3f(t[1], -TMAX, TMAX));
                t = t + 1.0f;
                t[0] = __builtin_amdgcn_rcpf(t[0]);
                t[1] = __builtin_amdgcn_rcpf(t[1]);
                const f32x2 v = y * t;
                const float v0 = v[0], v1 = v[1];
                o[i] = __builtin_bit_cast(unsigned, v0);
                o[i + 1] = __builtin_bit_cast(unsigned, v1);
            }
            __builtin_amdgcn_raw_buffer_store_b128(o, rsO, (int)(ooff + r * (NN * 4)), 0, 17);
        }
    }
    }
    STAMP(29);
}

}

extern "C" void kernel_launch(int rank, int nranks, ncclComm_t comm,
                              hipStream_t stream,
                              void* const* d_in, const int* in_sizes, int n_in,
                              void* d_out, int out_size,
                              const int* in_dtypes, int out_dtype,
                              void* const* d_out_all,
                              void* const* d_ws_all, size_t ws_bytes) {
    (void)comm; (void)in_dtypes; (void)out_dtype; (void)d_out_all;
    if (nranks != NR || (unsigned)rank >= (unsigned)NR || n_in != 4 || ws_bytes < WS_NEEDED ||
        in_sizes[0] != 4096 * KS || in_sizes[1] != KS * NN || out_size != MS * NN) {
        fprintf(stderr, "kernel_launch: unexpected problem shape\n");
        return;
    }
    Params p;
    p.x = (const int8_t*)d_in[0];
    p.w = (const int8_t*)d_in[1];
    p.sx = (const float*)d_in[2];
    p.sw = (const float*)d_in[3];
    p.out = (float*)d_out;
    for (int t = 0; t < NR; ++t) p.ws[t] = (char*)d_ws_all[(rank + t) % NR];
    p.rank = rank;
    p.pad = 0;
    hipLaunchKernelGGL(gemm_ag_kernel, dim3(NWG), dim3(NTHREADS), 0, stream, p);
    HIP_CHECK(hipGetLastError());
}
```

```cpp
#include <hip/hip_runtime.h>
#include <rccl/rccl.h>
#include <cstdio>
#include <cstdint>

#define HIP_CHECK(expr) do { hipError_t _e = (expr); if (_e != hipSuccess) { \
    fprintf(stderr, "HIP error %s at %s:%d\n", hipGetErrorString(_e), __FILE__, __LINE__); \
    return; } } while (0)

namespace {

#define GAS __attribute__((address_space(1)))
#define LAS __attribute__((address_space(3)))
typedef unsigned u32x4 __attribute__((ext_vector_type(4)));
typedef int i32x4 __attribute__((ext_vector_type(4)));
typedef float f32x2 __attribute__((ext_vector_type(2)));
typedef unsigned u32x32 __attribute__((ext_vector_type(32)));

constexpr int NR = 8;
constexpr int MS = 512;
constexpr int KS = 512;
constexpr int NN = 4096;
constexpr int NCH = 8;
constexpr int NWG = 256;
constexpr int NUNIT = NCH * 64;
constexpr int NCW = 10;
constexpr int W_LOAD = 10, W_SEND = 11;
constexpr int NWAVE = 12;
constexpr int NTHREADS = NWAVE * 64;
constexpr int NSEND = 32;
#define WIN 10
#define LDN 12
#define TDELAY 8
#define XWIN 1
constexpr unsigned NJOB = NUNIT;
static_assert(WIN >= 2 && WIN <= 14 && LDN >= 2 && LDN + WIN <= 27 && 2 * NSEND <= 64, "sender window");
constexpr unsigned TICKETS = NJOB - NSEND;

constexpr unsigned OFF_CNT = 0;
constexpr unsigned OFF_TMO = 256;
constexpr unsigned OFF_HEAD = 512;
constexpr unsigned OFF_FLAGT = 4096;
constexpr unsigned OFF_FLAGX = 8192;
constexpr unsigned OFF_FLAGW = 16384;
constexpr unsigned OFF_DATA = 1u << 20;
constexpr unsigned W_BYTES = 16u << 20;
constexpr unsigned X_BYTES = 2u << 20;
constexpr unsigned PAR_STRIDE = W_BYTES + X_BYTES;
constexpr size_t WS_NEEDED = (size_t)OFF_DATA + 2 * (size_t)PAR_STRIDE;

constexpr unsigned L_B = 0;
constexpr unsigned L_RING = 0;
constexpr unsigned L_SEQ = 131072;
constexpr unsigned L_SYNC = 131328;
constexpr unsigned S_CALL = L_SYNC, S_RING = L_SYNC + 8, S_TDONE = L_SYNC + 12;
constexpr unsigned S_DONE = L_SYNC + 32;
constexpr unsigned LDS_BYTES = L_SYNC + 256;

#define DEADLINE 100000000ull
constexpr unsigned long long DEADLINE_TICKS = DEADLINE;

struct Params {
    const int8_t* x;
    const int8_t* w;
    const float* sx;
    const float* sw;
    float* out;
    char* ws[NR];
    int rank;
    int pad;
};

#define STAMP(i) do {} while (0)
#define STAMP_DUMP() do {} while (0)
constexpr unsigned LDS_TOTAL = LDS_BYTES;

template <int N> __device__ __forceinline__ void waitvm() { asm volatile("s_waitcnt vmcnt(%0)" :: "n"(N) : "memory"); }
__device__ __forceinline__ void st16_wt(char* base, unsigned off, u32x4 v) {
    asm volatile("global_store_dwordx4 %0, %1, %2 sc0 sc1\n\ts_nop 1" :: "v"(off), "v"(v), "s"(base) : "memory");
}
__device__ __forceinline__ void glds64_sys(const void* gsrc, unsigned lds_dst) {
    unsigned keep;
    asm volatile("s_mov_b32 %0, m0\n\ts_mov_b32 m0, %2\n\ts_nop 0\n\t"
                 "global_load_lds_dwordx4 %1, off sc0 sc1\n\tglobal_load_lds_dwordx4 %1, off offset:1024 sc0 sc1\n\t"
                 "global_load_lds_dwordx4 %1, off offset:2048 sc0 sc1\n\tglobal_load_lds_dwordx4 %1, off offset:3072 sc0 sc1\n\t"
                 "s_mov_b32 m0, %0"
                 : "=&s"(keep) : "v"(gsrc), "s"(lds_dst) : "memory");
}
__device__ __forceinline__ void lds_rd4(u32x4 (&f)[4], unsigned addr) {
    asm volatile("ds_read_b128 %0, %4\n\tds_read_b128 %1, %4 offset:1024\n\tds_read_b128 %2, %4 offset:2048\n\t"
                 "ds_read_b128 %3, %4 offset:3072\n\ts_waitcnt lgkmcnt(0)"
                 : "=&v"(f[0]), "=&v"(f[1]), "=&v"(f[2]), "=&v"(f[3]) : "v"(addr) : "memory");
}
__device__ __forceinline__ void ticket_issue(unsigned& q, GAS unsigned* head) {
    unsigned zero = 0u, one = 1u;
    asm volatile("global_atomic_add %0, %1, %2, %3 sc0" : "=v"(q) : "v"(zero), "v"(one), "s"(head) : "memory");
}
__device__ __forceinline__ void lds_add(unsigned addr, unsigned v) {
    asm volatile("ds_add_u32 %0, %1" :: "v"(addr), "v"(v) : "memory");
}
__device__ __forceinline__ void lds_st(unsigned addr, unsigned v) {
    asm volatile("ds_write_b32 %0, %1" :: "v"(addr), "v"(v) : "memory");
}
__device__ __forceinline__ unsigned lds_ld(unsigned addr) {
    unsigned r;
    asm volatile("ds_read_b32 %0, %1\n\ts_waitcnt lgkmcnt(0)" : "=v"(r) : "v"(addr) : "memory");
    return r;
}
__device__ __forceinline__ unsigned lds_ld_u(unsigned addr) { return (unsigned)__builtin_amdgcn_readfirstlane(lds_ld(addr)); }
__device__ __forceinline__ void lds_wait_ge(unsigned addr, unsigned want) {
    while (lds_ld_u(addr) < want) __builtin_amdgcn_s_sleep(1);
}

__device__ __forceinline__ bool give_up(unsigned long long t0, unsigned spins, GAS unsigned* tmo, unsigned call) {
    bool g = __builtin_amdgcn_s_memrealtime() - t0 > DEADLINE_TICKS;
    if ((spins & 15u) == 15u)
        g |= (unsigned)__builtin_amdgcn_readfirstlane(
                 __hip_atomic_load(tmo, __ATOMIC_RELAXED, __HIP_MEMORY_SCOPE_AGENT)) == call;
    if (g) __hip_atomic_store(tmo, call, __ATOMIC_RELAXED, __HIP_MEMORY_SCOPE_AGENT);
    return g;
}
__device__ __forceinline__ void wait_flags(GAS unsigned* flag, unsigned want, GAS unsigned* tmo, unsigned call) {
    unsigned long long t0 = __builtin_amdgcn_s_memrealtime();
    for (unsigned spins = 0;; ++spins) {
        unsigned v = __hip_atomic_load(flag, __ATOMIC_RELAXED, __HIP_MEMORY_SCOPE_SYSTEM);
        if (__all((int)(v - want) >= 0)) break;
        if (give_up(t0, spins, tmo, call)) break;
        __builtin_amdgcn_s_sleep(4);
    }
}

__device__ __forceinline__ unsigned unit_order(unsigned u, unsigned rank) {
    return (u & ~63u) | ((u + 8u * rank) & 63u);
}

template <int N> __device__ __forceinline__ void waitvm_unless(unsigned skip) {
    asm volatile("s_cmp_lg_u32 %0, 0\n\ts_cbranch_scc1 .Lnowait%=\n\ts_waitcnt vmcnt(%1)\n.Lnowait%=:" :: "s"(skip), "n"(N) : "memory", "scc");
}
template <int N> __device__ __forceinline__ void waitvm_unless_n(unsigned skip, int n) {
    if (n == 0) waitvm_unless<N>(skip); else if (n == 1) waitvm_unless<N + 1>(skip); else if (n == 2) waitvm_unless<N + 2>(skip);
    else if (n == 3) waitvm_unless<N + 3>(skip); else if (n == 4) waitvm_unless<N + 4>(skip);
    else if (n == 5) waitvm_unless<N + 5>(skip); else waitvm_unless<N + 6>(skip);
}

__global__ void __launch_bounds__(NTHREADS, 1) gemm_ag_kernel(Params p) {
    __shared__ __attribute__((aligned(16))) char smem[LDS_TOTAL];
    const int tid = threadIdx.x;
    const int lane = tid & 63;
    const int wv = __builtin_amdgcn_readfirstlane(tid >> 6);
    const unsigned g = blockIdx.x;
    const unsigned lds0 = (unsigned)(size_t)(LAS char*)smem;
    const bool send_wg = g < NSEND;
    char* const myws = p.ws[0];
    GAS unsigned* const tmo = (GAS unsigned*)(myws + OFF_TMO);
    STAMP(0);

    const unsigned xrb = g >> 3;
    const unsigned xrel = (g + (g >> 3)) & 7u;
    u32x4 xv[NCH];
    unsigned tr[16];
    const unsigned tu = unit_order(2u * g + wv, p.rank);
    GAS unsigned* const head = (GAS unsigned*)(myws + OFF_HEAD);

    if (wv == W_SEND) {
        unsigned cs;
        {
            unsigned zero = 0u;
            asm volatile("global_load_dword %0, %1, %2 sc1" : "=v"(cs) : "v"(zero), "s"(myws + OFF_CNT) : "memory");
        }
        const __amdgpu_buffer_rsrc_t rs = __builtin_amdgcn_make_buffer_rsrc((void*)p.x, 0, 4096 * KS, 0x00020000);
        const unsigned d = (p.rank + xrel) & 7u;
#pragma unroll
        for (int j = 0; j < NCH; ++j)
            xv[j] = __builtin_amdgcn_raw_buffer_load_b128(
                rs, (int)((d * MS + xrb * 16u + (lane & 15)) * KS + j * 64u + (lane >> 4) * 16u), 0, 0);
        asm volatile("s_waitcnt vmcnt(0)" : "+v"(cs) :: "memory");
#pragma unroll
        for (int j = 0; j < NCH; ++j) asm volatile("" : "+v"(xv[j]));
        const unsigned scall = (unsigned)__builtin_amdgcn_readfirstlane(cs) / NWG + 1u;
        const unsigned xoff = OFF_DATA + (scall & 1u) * PAR_STRIDE + W_BYTES + (p.rank * 32u + xrb) * 8192u + lane * 16u;
#pragma unroll
        for (int j = 0; j <= XWIN; ++j) st16_wt(p.ws[xrel], xoff + j * 1024u, xv[j]);
    } else if (wv == W_LOAD) {
        unsigned c = 0;
        if (lane == 0) c = __hip_atomic_load((GAS unsigned*)(myws + OFF_CNT), __ATOMIC_RELAXED, __HIP_MEMORY_SCOPE_AGENT);
        if (lane < NCH) *(LAS unsigned*)(smem + L_SEQ + lane * 4) = 0u;
        if (lane >= 1 && lane < 16) *(LAS unsigned*)(smem + L_SYNC + lane * 4) = 0u;
        if (lane == 0) *(LAS unsigned*)(smem + S_CALL) = c / NWG + 1u;
        asm volatile("s_waitcnt lgkmcnt(0)" ::: "memory");
    } else if (wv < 2) {
        if (TDELAY) __builtin_amdgcn_s_sleep(TDELAY);
        const unsigned u = tu;
        const __amdgpu_buffer_rsrc_t rs = __builtin_amdgcn_make_buffer_rsrc((void*)p.w, 0, KS * NN, 0x00020000);
#pragma unroll
        for (int t = 0; t < 16; ++t)
            tr[t] = __builtin_amdgcn_raw_buffer_load_b32(
                rs, (int)(((u >> 6) * 64u + (lane >> 4) * 16u + t) * NN + (u & 63u) * 64u + (lane & 15) * 4u), 0, 0);
    }
    __builtin_amdgcn_s_barrier();
    asm volatile("" ::: "memory");
    const unsigned call = lds_ld_u(lds0 + S_CALL);
    const unsigned par = OFF_DATA + (call & 1u) * PAR_STRIDE;
    const unsigned ci = (g - NSEND) >> 3;
    const unsigned cp = (g & 7u) * 4u + ci / 7u;
    const unsigned rg = ci % 7u;
    const unsigned nrb = rg < 4u ? 5u : 4u;
    const unsigned rb0 = rg < 4u ? rg * 5u : 20u + (rg - 4u) * 4u;
    STAMP(1);

    if (wv == W_SEND) {
        __builtin_amdgcn_s_setprio(2);
        char* const xd = p.ws[xrel];
        const unsigned xoff = par + W_BYTES + (p.rank * 32u + xrb) * 8192u + lane * 16u;
        STAMP(2);
#pragma unroll
        for (int j = XWIN + 1; j < NCH; ++j) {
            waitvm<XWIN>();
            st16_wt(xd, xoff + j * 1024u, xv[j]);
        }
        GAS unsigned* const fx = (GAS unsigned*)(xd + OFF_FLAGX + xrb * 128u + p.rank * 4u);
        if (!send_wg) {
            waitvm<0>();
            if (lane == 0) __hip_atomic_store(fx, call, __ATOMIC_RELAXED, __HIP_MEMORY_SCOPE_SYSTEM);
            STAMP(3);
            STAMP_DUMP();
            return;
        }
        STAMP(3);
        char* peer = p.ws[0];
#pragma unroll
        for (int t = 1; t < NR; ++t) peer = ((lane & 7) == t) ? p.ws[t] : peer;
        char* dst[NR - 1];
#pragma unroll
        for (int t = 0; t < NR - 1; ++t) dst[t] = p.ws[1 + (t + g) % 7u];
        const unsigned qbase = (call - 1u) * TICKETS - 2u * NSEND;
        unsigned jcur = 2u * g;
        unsigned jnxt = 2u * g + 1u;
        const unsigned ring = (unsigned)__builtin_amdgcn_readfirstlane(lds0 + L_RING);
        const unsigned wbase = par + p.rank * (NUNIT * 4096u);
        const char* wloc = myws + wbase + lane * 16;
        GAS unsigned* fprev = fx;
        bool fact = lane == 0;
        lds_wait_ge(lds0 + S_RING, 2u);
        unsigned slot = 0;
#define JOB_UNIT(jb) unit_order((jb), p.rank)
#define JOB_FLAG(jb) ((GAS unsigned*)(peer + OFF_FLAGW + (JOB_UNIT(jb) * 8u + p.rank) * 4u))
#pragma unroll 1
        for (unsigned nth = 0;; ++nth) {
            const unsigned last = (unsigned)__builtin_amdgcn_readfirstlane(jnxt >= NJOB ? 1 : 0);
            const char* nsrc = wloc + JOB_UNIT(last ? jcur : jnxt) * 4096u;
            const unsigned nslot = ring + (slot ^ 1u) * 4096u;
            unsigned qn = 0;
            u32x4 f[4];
            lds_rd4(f, ring + slot * 4096u + lane * 16u);
            const unsigned woff = wbase + JOB_UNIT(jcur) * 4096u + lane * 16u;
            STAMP(4 + (jcur >> 6));
#pragma unroll
            for (int k = 0; k < 28; ++k) {
                st16_wt(dst[k % 7], woff + (k / 7) * 1024u, f[k / 7]);
                const int n = ((WIN - 1 >= k - WIN && WIN - 1 <= k - 1) ? 1 : 0) + ((LDN >= k - WIN && LDN <= k - 1) ? 5 : 0);
                waitvm_unless_n<WIN>(last, n);
                if (k == WIN - 1) {
                    waitvm<WIN>();
                    if (fact) __hip_atomic_store(fprev, call, __ATOMIC_RELAXED, __HIP_MEMORY_SCOPE_SYSTEM);
                }
                if (k == LDN && !last) {
                    if (nth == 1) lds_wait_ge(lds0 + S_TDONE, 1u);
                    if (lane == 0) ticket_issue(qn, head);
                    if (nth != 0) glds64_sys(nsrc, nslot);
                }
            }
            fprev = JOB_FLAG(jcur);
            fact = lane >= 1 && lane < NR;
            if (last) break;
            waitvm<(27 - LDN) + (WIN - 1 > LDN ? 1 : 0)>();
            asm volatile("" : "+v"(qn));
            jcur = jnxt;
            jnxt = (unsigned)__builtin_amdgcn_readfirstlane(qn) - qbase;
            slot ^= 1u;
        }
        STAMP(12);
        waitvm<0>();
        STAMP(13);
        if (lane >= 1 && lane < NR) __hip_atomic_store(fprev, call, __ATOMIC_RELAXED, __HIP_MEMORY_SCOPE_SYSTEM);
        STAMP_DUMP();
        return;
    }

    if (wv == W_LOAD) {
        __builtin_amdgcn_s_setprio(3);
        if (send_wg) {
            const char* ft = myws + OFF_FLAGT + lane * 16;
            unsigned long long t0 = __builtin_amdgcn_s_memrealtime();
            for (unsigned spins = 0;; ++spins) {
                u32x4 a, b;
                asm volatile("global_load_dwordx4 %0, %2, off sc0 sc1\n\tglobal_load_dwordx4 %1, %2, off offset:1024 sc0 sc1\n\t"
                             "s_waitcnt vmcnt(0)" : "=&v"(a), "=&v"(b) : "v"(ft) : "memory");
                bool ok = true;
#pragma unroll
                for (int e = 0; e < 4; ++e) ok = ok && (int)(a[e] - call) >= 0 && (int)(b[e] - call) >= 0;
                if (__all(ok)) break;
                if (give_up(t0, spins, tmo, call)) break;
                __builtin_amdgcn_s_sleep(2);
            }
            if (lane == 0) lds_st(lds0 + S_TDONE, 1u);
            STAMP(12);
            if (lane == 0)
                (void)__hip_atomic_fetch_add((GAS unsigned*)(myws + OFF_CNT), 1u, __ATOMIC_RELAXED, __HIP_MEMORY_SCOPE_AGENT);
            STAMP_DUMP();
            return;
        }
        const char* wsrc = myws + par + cp * 8192u + lane * 16;
        const unsigned lbase = (unsigned)__builtin_amdgcn_readfirstlane(lds0 + L_B);
        const unsigned nmfma = 2u * nrb;
        unsigned cs0 = 0, cs1 = 1;
        bool act0 = false, act1 = false;
        unsigned pub0 = 0, pub1 = 0, tmo_now = 0;
        unsigned pending = 0, issued = 0;
        bool counted = false;
        const unsigned long long t0 = __builtin_amdgcn_s_memrealtime();
        for (unsigned spins = 0;; ++spins) {
            if (!act0 && cs0 < NCH && (cs0 < 2 || lds_ld_u(lds0 + S_DONE + (cs0 - 2) * 4u) >= nmfma)) {
                act0 = true; pub0 = 0; pending |= 0xffffu; STAMP(16 + cs0);
            }
            if (!act1 && cs1 < NCH && (cs1 < 2 || lds_ld_u(lds0 + S_DONE + (cs1 - 2) * 4u) >= nmfma)) {
                act1 = true; pub1 = 0; pending |= 0xffff0000u; STAMP(16 + cs1);
            }
            unsigned ready = 0;
            if (pending) {
                const unsigned cl = ((lane & 16) ? cs1 : cs0) & 7u;
                const unsigned fo = OFF_FLAGW + ((cl * 64u + cp * 2u + (lane & 1)) * 8u + ((lane >> 1) & 7)) * 4u;
                unsigned v = __hip_atomic_load((GAS unsigned*)(myws + fo), __ATOMIC_RELAXED, __HIP_MEMORY_SCOPE_SYSTEM);
                ready = (unsigned)__ballot((int)(v - call) >= 0) & pending;
                if (!ready && give_up(t0, spins, tmo, call)) ready = pending;
            }
            waitvm<0>();
            unsigned both = issued & (issued >> 1) & 0x55555555u;
            both = (both | (both >> 1)) & 0x33333333u;
            both = (both | (both >> 2)) & 0x0f0f0f0fu;
            both = (both | (both >> 4)) & 0x00ff00ffu;
            const unsigned n0 = both & 0xffu, n1 = both >> 16;
            const unsigned badbit = (!pending && cs0 >= NCH - 2 && cs1 >= NCH - 1 && (act0 || cs0 >= NCH) && (act1 || cs1 >= NCH) &&
                                     tmo_now == call) ? 0x100u : 0u;
            if (act0 && n0 != pub0) {
                pub0 = n0;
                if (lane == 0) lds_st(lds0 + L_SEQ + cs0 * 4u, pub0 | badbit);
                if (pub0 == 0xffu) { STAMP(4 + cs0); act0 = false; cs0 += 2; issued &= 0xffff0000u; }
            }
            if (act1 && n1 != pub1) {
                pub1 = n1;
                if (lane == 0) lds_st(lds0 + L_SEQ + cs1 * 4u, pub1 | badbit);
                if (pub1 == 0xffu) { STAMP(4 + cs1); act1 = false; cs1 += 2; issued &= 0xffffu; }
            }
            if (cs0 >= NCH && cs1 >= NCH) break;
            if (!counted && cs0 >= 2) {
                counted = true;
                if (lane == 0)
                    (void)__hip_atomic_fetch_add((GAS unsigned*)(myws + OFF_CNT), 1u, __ATOMIC_RELAXED, __HIP_MEMORY_SCOPE_AGENT);
            }
            if (!ready) { if (cs1 < NCH - 1) __builtin_amdgcn_s_sleep(1); continue; }
            pending &= ~ready;
            issued |= ready;
            while (ready) {
                const unsigned b = __builtin_ctz(ready);
                ready &= ready - 1;
                const unsigned c = b >= 16 ? cs1 : cs0, q = (b >> 1) & 7u, h = b & 1u;
                glds64_sys(wsrc + (q * NCH + c) * (64u * 4096u) + h * 4096u, lbase + (c & 1u) * 65536u + q * 8192u + h * 4096u);
            }
            if (!pending && cs0 >= NCH - 2 && cs1 >= NCH - 1)
                tmo_now = (unsigned)__builtin_amdgcn_readfirstlane(__hip_atomic_load(tmo, __ATOMIC_RELAXED, __HIP_MEMORY_SCOPE_AGENT));
        }
        STAMP_DUMP();
        return;
    }

    if (wv < 2) {
        const unsigned u = tu;
        u32x4 f[4];
#pragma unroll
        for (int q = 0; q < 4; ++q) {
            unsigned t0 = __builtin_amdgcn_perm(tr[4 * q + 1], tr[4 * q + 0], 0x05010400u);
            unsigned t1 = __builtin_amdgcn_perm(tr[4 * q + 1], tr[4 * q + 0], 0x07030602u);
            unsigned t2 = __builtin_amdgcn_perm(tr[4 * q + 3], tr[4 * q + 2], 0x05010400u);
            unsigned t3 = __builtin_amdgcn_perm(tr[4 * q + 3], tr[4 * q + 2], 0x07030602u);
            f[0][q] = __builtin_amdgcn_perm(t2, t0, 0x05040100u);
            f[1][q] = __builtin_amdgcn_perm(t2, t0, 0x07060302u);
            f[2][q] = __builtin_amdgcn_perm(t3, t1, 0x05040100u);
            f[3][q] = __builtin_amdgcn_perm(t3, t1, 0x07060302u);
        }
        const unsigned woff = par + (p.rank * NUNIT + u) * 4096u + lane * 16u;
#pragma unroll
        for (int i = 0; i < 4; ++i) st16_wt(myws, woff + i * 1024u, f[i]);
        if (send_wg) {
#pragma unroll
            for (int i = 0; i < 4; ++i) *(LAS u32x4*)(smem + L_RING + wv * 4096 + i * 1024 + lane * 16) = f[i];
            asm volatile("s_waitcnt lgkmcnt(0)" ::: "memory");
            if (lane == 0) lds_add(lds0 + S_RING, 1u);
        }
        waitvm<0>();
        if (lane == 0) {
            __hip_atomic_store((GAS unsigned*)(myws + OFF_FLAGT + u * 4u), call, __ATOMIC_RELAXED, __HIP_MEMORY_SCOPE_SYSTEM);
            __hip_atomic_store((GAS unsigned*)(myws + OFF_FLAGW + (u * 8u + p.rank) * 4u), call, __ATOMIC_RELAXED, __HIP_MEMORY_SCOPE_SYSTEM);
        }
    }
    if (send_wg || (unsigned)wv >= 2u * nrb) { STAMP_DUMP(); return; }
    float scale = p.sx[0] * p.sw[0];
    const unsigned half = (unsigned)wv >= nrb ? 1u : 0u;
    const unsigned rb = rb0 + (unsigned)wv - half * nrb;
    const unsigned nj = cp * 2u + half;
    const __amdgpu_buffer_rsrc_t rsX =
        __builtin_amdgcn_make_buffer_rsrc((void*)(myws + par + W_BYTES), 0, (int)X_BYTES, 0x00020000);

    float c2 = scale * -1.4426950408889634f;
    char* const outp = (char*)p.out;
    unsigned ooff = ((rb * 16u + (lane >> 4) * 4u) * NN + nj * 64u + (lane & 15) * 4u) * 4u;
    asm volatile("" : "+v"(scale), "+v"(c2), "+v"(ooff) : "s"(outp));

    i32x4 acc[4] = {};
    unsigned bad = 0;
    int npass = 2;
    asm volatile("" : "+s"(npass));
#pragma unroll 1
    for (int pass = 0; pass < npass; ++pass) {
    if (pass != 0) {
        wait_flags((GAS unsigned*)(myws + OFF_FLAGX + rb * 128u + (lane & 7) * 4u), call, tmo, call);

        STAMP(2);
#pragma unroll 1
        for (int j = 0; j < NCH; ++j) {
            u32x32 a;
#pragma unroll
            for (int q = 0; q < NR; ++q) {
                const u32x4 t = __builtin_amdgcn_raw_buffer_load_b128(rsX, (int)(((q * 32u + rb) * 8u + j) * 1024u + lane * 16u), 0, 17);
                a[4 * q] = t[0]; a[4 * q + 1] = t[1]; a[4 * q + 2] = t[2]; a[4 * q + 3] = t[3];
            }
            const char* lb = smem + L_B + (j & 1) * 65536 + half * 4096 + lane * 16;
            unsigned seen = 0, m = 0;
#pragma unroll 1
            for (int left = NR; left > 0;) {
                if (m == 0u) {
                    const unsigned word = lds_ld_u(lds0 + L_SEQ + j * 4u);
                    bad |= word >> 8;
                    m = word & 0xffu & ~seen;
                    if (m == 0u) { __builtin_amdgcn_s_sleep(1); continue; }
                    if (seen == 0) STAMP(4 + j * 3);
                    seen |= m;
                }
                const unsigned q = __builtin_ctz(m);
                m &= m - 1;
                --left;
                i32x4 b[4];
#pragma unroll
                for (int i = 0; i < 4; ++i) b[i] = *(const LAS i32x4*)(lb + q * 8192u + i * 1024);
                const u32x4 av = {a[4 * q], a[4 * q + 1], a[4 * q + 2], a[4 * q + 3]};
#pragma unroll
                for (int i = 0; i < 4; ++i)
                    acc[i] = __builtin_amdgcn_mfma_i32_16x16x64_i8(__builtin_bit_cast(i32x4, av), b[i], acc[i], 0, 0, 0);
            }
            STAMP(5 + j * 3);
            if (j < NCH - 2 && lane == 0) lds_add(lds0 + S_DONE + j * 4u, 1u);
        }
    }
    STAMP(28);
    const __amdgpu_buffer_rsrc_t rsO = __builtin_amdgcn_make_buffer_rsrc((void*)outp, 0, pass != 0 ? MS * NN * 4 : 0, 0x00020000);
    if (__builtin_expect(bad != 0u, 0)) {
        const u32x4 nan4 = {0x7fc00000u, 0x7fc00000u, 0x7fc00000u, 0x7fc00000u};
#pragma unroll
        for (int r = 0; r < 4; ++r) __builtin_amdgcn_raw_buffer_store_b128(nan4, rsO, (int)(ooff + r * (NN * 4)), 0, 17);
    } else {
        constexpr float TMAX = 60.0f * 1.4426950408889634f;
#pragma unroll
        for (int r = 0; r < 4; ++r) {
            u32x4 o;
#pragma unroll
            for (int i = 0; i < 4; i += 2) {
                const f32x2 a = {(float)acc[i][r], (float)acc[i + 1][r]};
                const f32x2 y = a * scale;
                f32x2 t = a * c2;
                t[0] = __builtin_amdgcn_exp2f(__builtin_amdgcn_fmed3f(t[0], -TMAX, TMAX));
                t[1] = __builtin_amdgcn_exp2f(__builtin_amdgcn_fmed3f(t[1], -TMAX, TMAX));
                t = t + 1.0f;
                t[0] = __builtin_amdgcn_rcpf(t[0]);
                t[1] = __builtin_amdgcn_rcpf(t[1]);
                const f32x2 v = y * t;
                const float v0 = v[0], v1 = v[1];
                o[i] = __builtin_bit_cast(unsigned, v0);
                o[i + 1] = __builtin_bit_cast(unsigned, v1);
            }
            __builtin_amdgcn_raw_buffer_store_b128(o, rsO, (int)(ooff + r * (NN * 4)), 0, 17);
        }
    }
    }
    STAMP(29);
}

}

extern "C" void kernel_launch(int rank, int nranks, ncclComm_t comm,
                              hipStream_t stream,
                              void* const* d_in, const int* in_sizes, int n_in,
                              void* d_out, int out_size,
                              const int* in_dtypes, int out_dtype,
                              void* const* d_out_all,
                              void* const* d_ws_all, size_t ws_bytes) {
    (void)comm; (void)in_dtypes; (void)out_dtype; (void)d_out_all;
    if (nranks != NR || (unsigned)rank >= (unsigned)NR || n_in != 4 || ws_bytes < WS_NEEDED ||
        in_sizes[0] != 4096 * KS || in_sizes[1] != KS * NN || out_size != MS * NN) {
        fprintf(stderr, "kernel_launch: unexpected problem shape\n");
        return;
    }
    Params p;
    p.x = (const int8_t*)d_in[0];
    p.w = (const int8_t*)d_in[1];
    p.sx = (const float*)d_in[2];
    p.sw = (const float*)d_in[3];
    p.out = (float*)d_out;
    for (int t = 0; t < NR; ++t) p.ws[t] = (char*)d_ws_all[(rank + t) % NR];
    p.rank = rank;
    p.pad = 0;
    hipLaunchKernelGGL(gemm_ag_kernel, dim3(NWG), dim3(NTHREADS), 0, stream, p);
    HIP_CHECK(hipGetLastError());
}
```
